```python
import numpy as np

kernel_name = "gpu_gemm_a2a_m2048_k2048_n2048_bf16_none"


ABSMAX_RTOL = 0.008

COMPUTE_DTYPE = "bfloat16"
EPILOGUE = "none"
IN_QUANT = None
OUT_QUANT = None
IN_DTYPES = {"x": "bfloat16", "w_mat": "bfloat16"}
WORLD_SIZE = 8

def _to_bf16(x):
    u = np.ascontiguousarray(x, dtype=np.float32).view(np.uint32)
    return ((u + 0x7FFF + ((u >> 16) & 1)) & 0xFFFF0000).view(np.float32)

def setup_inputs():

    x = _to_bf16(np.random.randn(2048, 2048).astype(np.float32))
    w_mat = _to_bf16((np.random.randn(2048, 2048) / np.sqrt(2048)).astype(np.float32))
    return {"x": x, "w_mat": w_mat}

def _epilogue(y):
    return y

def reference(x, w_mat):
    y = (x @ w_mat).astype(np.float32)
    return _epilogue(y)
```

```cpp
#include <hip/hip_runtime.h>
#include <rccl/rccl.h>
#include <cstdio>
#include <cstdint>

#define HIP_CHECK(expr) do { hipError_t _e = (expr); if (_e != hipSuccess) { \
    fprintf(stderr, "HIP error %s at %s:%d\n", hipGetErrorString(_e), __FILE__, __LINE__); \
    return; } } while (0)

#define LAS __attribute__((address_space(3)))
#define GAS __attribute__((address_space(1)))

typedef short v4s __attribute__((ext_vector_type(4)));
typedef short v8s __attribute__((ext_vector_type(8)));
typedef __bf16 v8bf __attribute__((ext_vector_type(8)));
typedef float v4f __attribute__((ext_vector_type(4)));
typedef unsigned u32x4 __attribute__((ext_vector_type(4)));

namespace {

constexpr int NRANK = 8;
constexpr int MLOC = 256;
constexpr int KDIM = 2048, NDIM = 2048;
constexpr int NLOC = NDIM / NRANK;
constexpr int BM = 32, BN = 64, BK = 64;
constexpr int NWAVE = 4;
constexpr int NST = KDIM / BK / NWAVE;
constexpr int A_BYTES = BM * BK * 2;
constexpr int B_BYTES = BK * BN * 2;
constexpr int ST_BYTES = A_BYTES + B_BYTES;
constexpr int RING = 3;
constexpr int WAVE_LDS = RING * ST_BYTES;
constexpr int LDS_BYTES = NWAVE * WAVE_LDS;
constexpr int NWG = (MLOC / BM) * (NDIM / BN);
constexpr int PROW = 272;

constexpr size_t WS_LINE = 128;
constexpr size_t WS_CNT = 0;
constexpr size_t WS_FLAG = WS_CNT + NWG * WS_LINE;
constexpr size_t WS_TMO = WS_FLAG + NRANK * 32 * WS_LINE;
constexpr size_t WS_END = WS_TMO + WS_LINE;

constexpr unsigned long long DEADLINE_TICKS = 40000000ull;

struct Params {
    const void* x;
    const void* w;
    void* out[NRANK];
    void* ws[NRANK];
    int rank;
    int pad;
};

#define WAIT_VM(n) asm volatile("s_waitcnt vmcnt(%0)" :: "n"(n) : "memory")
#define WAIT_LGKM0() asm volatile("s_waitcnt lgkmcnt(0)" ::: "memory")

template <typename R>
__device__ __forceinline__ void dma16(unsigned lds_addr, unsigned voff, R srd, unsigned soff) {
    asm volatile("s_mov_b32 m0, %0\n\ts_nop 0\n\tbuffer_load_dwordx4 %1, %2, %3 offen lds"
                 :: "s"(lds_addr), "v"(voff), "s"(srd), "s"(soff) : "memory");
}

__device__ __forceinline__ unsigned cvt_pk_bf16(float lo, float hi) {
    typedef __bf16 v2bf __attribute__((ext_vector_type(2)));
    typedef float v2f __attribute__((ext_vector_type(2)));
    v2f f = {lo, hi};
    return __builtin_bit_cast(unsigned, __builtin_convertvector(f, v2bf));
}

__global__ void __launch_bounds__(256, 1) gemm_a2a(Params p) {
    __shared__ __attribute__((aligned(1024))) char smem[LDS_BYTES];
    const unsigned tid = threadIdx.x;
    const unsigned lane = tid & 63;
    const unsigned wave = __builtin_amdgcn_readfirstlane(tid >> 6);
    const unsigned bid = blockIdx.x;
    const unsigned dest = bid & 7;
    const unsigned tile = bid >> 3;
    const unsigned m0 = (tile >> 2) * BM;
    const unsigned nl0 = (tile & 3) * BN;
    const unsigned n0 = dest * NLOC + nl0;

    char* myws = (char*)p.ws[p.rank];
    unsigned call = 0;
    if (tid == 0)
        call = __hip_atomic_fetch_add((GAS unsigned*)(myws + WS_CNT + bid * WS_LINE), 1u, __ATOMIC_RELAXED,
                                      __HIP_MEMORY_SCOPE_AGENT) + 1u;

    const auto srdX = __builtin_amdgcn_make_buffer_rsrc((void*)p.x, 0, MLOC * KDIM * 2, 0x00020000);
    const auto srdW = __builtin_amdgcn_make_buffer_rsrc((void*)p.w, 0, KDIM * NDIM * 2, 0x00020000);

    const unsigned ldsw = (unsigned)(size_t)(LAS char*)smem + wave * WAVE_LDS;
    const unsigned rsub = lane >> 3, pos = lane & 7;
    const unsigned chA = pos ^ (lane >> 4);
    const unsigned voffA0 = (m0 + rsub) * (KDIM * 2) + chA * 16;
    const unsigned voffA1 = (m0 + rsub) * (KDIM * 2) + (chA ^ 4) * 16;
    const unsigned chB = pos ^ (((lane >> 4) & 1) << 1);
    const unsigned voffB0 = rsub * (NDIM * 2) + n0 * 2 + chB * 16;
    const unsigned voffB1 = rsub * (NDIM * 2) + n0 * 2 + (chB ^ 4) * 16;

    auto issue = [&](int s) {
        const unsigned k0 = (s * NWAVE + wave) * BK;
        const unsigned dst = ldsw + (s % RING) * ST_BYTES;
#pragma unroll
        for (int i = 0; i < 4; ++i)
            dma16(dst + i * 1024, (i & 1) ? voffA1 : voffA0, srdX, k0 * 2 + i * 8 * (KDIM * 2));
#pragma unroll
        for (int i = 0; i < 8; ++i)
            dma16(dst + A_BYTES + i * 1024, (i & 1) ? voffB1 : voffB0, srdW, (k0 + i * 8) * (NDIM * 2));
    };

    LAS char* wbase = (LAS char*)smem + wave * WAVE_LDS;
    const unsigned fr = lane & 15, fg = lane >> 4;
    const unsigned rdA0 = fr * 128 + ((fg ^ ((lane >> 1) & 7)) * 16);
    const unsigned q = fr >> 2, pp = lane & 3;
    const unsigned fB = (q >> 1) | ((fg & 1) << 1);
    const unsigned rdB = (8 * fg + q) * 128 + fB * 32 + (pp >> 1) * 16 + (pp & 1) * 8;

    v4f acc[2][4];
#pragma unroll
    for (int mi = 0; mi < 2; ++mi)
#pragma unroll
        for (int ni = 0; ni < 4; ++ni) acc[mi][ni] = (v4f){0.f, 0.f, 0.f, 0.f};

    issue(0); issue(1); issue(2);

#pragma unroll
    for (int s = 0; s < NST; ++s) {
        if (s + 2 < NST) WAIT_VM(24); else if (s + 1 < NST) WAIT_VM(12); else WAIT_VM(0);
        LAS char* st = wbase + (s % RING) * ST_BYTES;
        v8s af[2][2], bf[4][2];
#pragma unroll
        for (int ks = 0; ks < 2; ++ks) {
#pragma unroll
            for (int mi = 0; mi < 2; ++mi)
                af[mi][ks] = *(LAS v8s*)(st + ((rdA0 ^ (ks * 64)) + mi * 2048));
#pragma unroll
            for (int ni = 0; ni < 4; ++ni) {
                LAS char* b = st + A_BYTES + ks * 4096 + (rdB ^ (ni * 32));
                v4s lo = __builtin_amdgcn_ds_read_tr16_b64_v4i16((LAS v4s*)b);
                v4s hi = __builtin_amdgcn_ds_read_tr16_b64_v4i16((LAS v4s*)(b + 512));
                bf[ni][ks] = __builtin_shufflevector(lo, hi, 0, 1, 2, 3, 4, 5, 6, 7);
            }
        }
        WAIT_LGKM0();
        if (s + RING < NST) issue(s + RING);
#pragma unroll
        for (int ks = 0; ks < 2; ++ks)
#pragma unroll
            for (int mi = 0; mi < 2; ++mi)
#pragma unroll
                for (int ni = 0; ni < 4; ++ni)
                    acc[mi][ni] = __builtin_amdgcn_mfma_f32_16x16x32_bf16(
                        __builtin_bit_cast(v8bf, bf[ni][ks]), __builtin_bit_cast(v8bf, af[mi][ks]), acc[mi][ni], 0, 0, 0);
    }

#pragma unroll
    for (int mi = 0; mi < 2; ++mi)
#pragma unroll
        for (int ni = 0; ni < 4; ++ni)
            *(LAS v4f*)(wbase + (16 * mi + fr) * PROW + (16 * ni + 4 * fg) * 4) = acc[mi][ni];
    __syncthreads();
    const unsigned orow = tid >> 3, oc8 = tid & 7;
    v4f s0 = {0.f, 0.f, 0.f, 0.f}, s1 = s0;
#pragma unroll
    for (int w = 0; w < NWAVE; ++w) {
        LAS char* src = (LAS char*)smem + w * WAVE_LDS + orow * PROW + oc8 * 32;
        s0 += *(LAS v4f*)src;
        s1 += *(LAS v4f*)(src + 16);
    }
    u32x4 o;
    o.x = cvt_pk_bf16(s0[0], s0[1]); o.y = cvt_pk_bf16(s0[2], s0[3]);
    o.z = cvt_pk_bf16(s1[0], s1[1]); o.w = cvt_pk_bf16(s1[2], s1[3]);
    char* dst = (char*)p.out[dest] + ((size_t)(p.rank * MLOC + m0 + orow) * NLOC + nl0 + oc8 * 8) * 2;
    asm volatile("global_store_dwordx4 %0, %1, off sc0 sc1\n\ts_nop 1" :: "v"(dst), "v"(o) : "memory");
    asm volatile("s_waitcnt vmcnt(0)" ::: "memory");
    __syncthreads();

    if (tid == 0 && dest != (unsigned)p.rank) {
        GAS unsigned* flag = (GAS unsigned*)((char*)p.ws[dest] + WS_FLAG + (p.rank * 32 + tile) * WS_LINE);
        __hip_atomic_store(flag, call, __ATOMIC_RELAXED, __HIP_MEMORY_SCOPE_SYSTEM);
        GAS unsigned* in = (GAS unsigned*)(myws + WS_FLAG + (dest * 32 + tile) * WS_LINE);
        GAS unsigned* tmo = (GAS unsigned*)(myws + WS_TMO);
        const unsigned long long t0 = __builtin_amdgcn_s_memrealtime();
        unsigned spins = 0;
        bool bad = false;
        while (__hip_atomic_load(in, __ATOMIC_RELAXED, __HIP_MEMORY_SCOPE_SYSTEM) < call) {
            if ((++spins & 63u) == 0u) {
                if (__hip_atomic_load(tmo, __ATOMIC_RELAXED, __HIP_MEMORY_SCOPE_AGENT) == call ||
                    __builtin_amdgcn_s_memrealtime() - t0 > DEADLINE_TICKS) {
                    __hip_atomic_store(tmo, call, __ATOMIC_RELAXED, __HIP_MEMORY_SCOPE_AGENT);
                    bad = true;
                    break;
                }
            }
            __builtin_amdgcn_s_sleep(1);
        }
        if (bad) {
            GAS unsigned short* e = (GAS unsigned short*)p.out[p.rank] + (size_t)(p.rank * MLOC + m0) * NLOC + nl0;
            __hip_atomic_store(e, (unsigned short)0x7FC0, __ATOMIC_RELAXED, __HIP_MEMORY_SCOPE_SYSTEM);
        }
    }
}

}

extern "C" void kernel_launch(int rank, int nranks, ncclComm_t comm, hipStream_t stream,
                              void* const* d_in, const int* in_sizes, int n_in,
                              void* d_out, int out_size,
                              const int* in_dtypes, int out_dtype,
                              void* const* d_out_all, void* const* d_ws_all, size_t ws_bytes) {
    (void)comm; (void)in_dtypes; (void)out_dtype; (void)d_out;
    if (nranks != NRANK || (unsigned)rank >= (unsigned)NRANK || n_in != 2 || in_sizes[0] != MLOC * KDIM ||
        in_sizes[1] != KDIM * NDIM || out_size != NDIM * NLOC || ws_bytes < WS_END) {
        fprintf(stderr, "kernel_launch: unexpected shapes\n");
        return;
    }
    Params p;
    p.x = d_in[0];
    p.w = d_in[1];
    for (int i = 0; i < NRANK; ++i) { p.out[i] = d_out_all[i]; p.ws[i] = d_ws_all[i]; }
    p.rank = rank;
    p.pad = 0;
    hipLaunchKernelGGL(gemm_a2a, dim3(NWG), dim3(256), 0, stream, p);
    HIP_CHECK(hipGetLastError());
}
```

```cpp
#include <hip/hip_runtime.h>
#include <rccl/rccl.h>
#include <cstdio>
#include <cstdint>

#define HIP_CHECK(expr) do { hipError_t _e = (expr); if (_e != hipSuccess) { \
    fprintf(stderr, "HIP error %s at %s:%d\n", hipGetErrorString(_e), __FILE__, __LINE__); \
    return; } } while (0)

#define LAS __attribute__((address_space(3)))
#define GAS __attribute__((address_space(1)))

namespace {

constexpr int NRANK = 8;
constexpr int MLOC = 256;
constexpr int KDIM = 2048, NDIM = 2048;
constexpr int NLOC = NDIM / NRANK;
constexpr int BM = 32, BN = 64;
constexpr int SLOT = 10240;
constexpr int WAVE_LDS = 4 * SLOT;
constexpr int LDS_BYTES = 4 * WAVE_LDS;
constexpr int NWG = (MLOC / BM) * (NDIM / BN);

constexpr size_t WS_LINE = 128;
constexpr size_t WS_CNT = 0;
constexpr size_t WS_FLAG = WS_CNT + NWG * WS_LINE;
constexpr size_t WS_TMO = WS_FLAG + NRANK * 32 * WS_LINE;
constexpr size_t WS_END = WS_TMO + WS_LINE;

constexpr unsigned long long DEADLINE_TICKS = 40000000ull;
static_assert(DEADLINE_TICKS == 40000000ull, "the asm wait loop holds this value as a literal");


struct Params {
    const void* x;
    const void* w;
    char* myws;
    char* myout;
    int rank;
    int pad;
    void* out[NRANK];
    void* ws[NRANK];
};

__global__ void __launch_bounds__(256, 1) gemm_a2a(Params p) {
    __shared__ __attribute__((aligned(1024))) char smem[LDS_BYTES];
    const unsigned tid = threadIdx.x;
    const unsigned lane = tid & 63;
    const unsigned wave = __builtin_amdgcn_readfirstlane(tid >> 6);
    const unsigned bid = blockIdx.x;
    const unsigned tile = bid >> 3;
    const unsigned ti = tile >> 2, tg = tile & 3;
    const unsigned dest = (bid - 2 * tg) & 7;
    const unsigned m0 = ti * BM;
    const unsigned nl0 = tg * BN;
    const unsigned n0 = dest * NLOC + nl0;

    const unsigned rsub = lane >> 3, pos = lane & 7;
    const unsigned chA = pos ^ (lane >> 4);
    unsigned vA0 = (m0 + rsub) * (KDIM * 2) + chA * 16 + wave * 128;
    unsigned vA1 = (m0 + 8 + rsub) * (KDIM * 2) + (chA ^ 4) * 16 + wave * 128 - 1024;
    const unsigned chB = pos ^ (((lane >> 4) & 1) << 1);
    const unsigned vBb = rsub * (NDIM * 2) + n0 * 2 + wave * (64 * NDIM * 2);
    unsigned vB0 = vBb + chB * 16;
    unsigned vB1 = vBb + (chB ^ 4) * 16 + 8 * (NDIM * 2) - 1024;
    unsigned vB2 = vB0 + 16 * (NDIM * 2) - 2048;
    unsigned vB3 = vB1 + 16 * (NDIM * 2) - 2048;
    unsigned rot3 = (3 * ((ti + tg) % 3)) & 7;
    asm volatile("" : "+v"(vA0), "+v"(vA1), "+v"(vB0), "+v"(vB1), "+v"(vB2), "+v"(vB3), "+s"(rot3));
    const auto srdX = __builtin_amdgcn_make_buffer_rsrc((void*)p.x, 0, MLOC * KDIM * 2, 0x00020000);
    const auto srdW = __builtin_amdgcn_make_buffer_rsrc((void*)p.w, 0, KDIM * NDIM * 2, 0x00020000);
    const unsigned ldsw = (unsigned)(size_t)(LAS char*)smem + wave * WAVE_LDS;

    const bool slowcls = (((((unsigned)(size_t)p.w) + n0 * 2) >> 7) & 7) == 3;
    const unsigned rot2 = slowcls ? ti : rot3;
    const unsigned rotW = rot2 << 20, rotX = rot2 * 512;
    asm volatile(
        "s_add_u32 m0, %[ldsw], 8192\n\t"
        "s_add_u32 s61, %[rotX], 0\n\t"
        "s_and_b32 s61, s61, 4095\n\t"
        "buffer_load_dwordx4 %[vA0], %[srdX], s61 offen lds\n\t"
        "buffer_load_dwordx4 %[vA1], %[srdX], s61 offen offset:1024 lds\n\t"
        "s_add_u32 m0, %[ldsw], 0\n\t"
        "s_add_u32 s60, %[rotW], 0\n\t"
        "s_and_b32 s60, s60, 0x7fffff\n\t"
        "buffer_load_dwordx4 %[vB0], %[srdW], s60 offen lds\n\t"
        "buffer_load_dwordx4 %[vB1], %[srdW], s60 offen offset:1024 lds\n\t"
        "buffer_load_dwordx4 %[vB2], %[srdW], s60 offen offset:2048 lds\n\t"
        "buffer_load_dwordx4 %[vB3], %[srdW], s60 offen offset:3072 lds\n\t"
        "s_add_u32 m0, %[ldsw], 4096\n\t"
        "s_add_u32 s61, s60, 131072\n\t"
        "buffer_load_dwordx4 %[vB0], %[srdW], s61 offen lds\n\t"
        "buffer_load_dwordx4 %[vB1], %[srdW], s61 offen offset:1024 lds\n\t"
        "buffer_load_dwordx4 %[vB2], %[srdW], s61 offen offset:2048 lds\n\t"
        "buffer_load_dwordx4 %[vB3], %[srdW], s61 offen offset:3072 lds\n\t"
        "s_add_u32 m0, %[ldsw], 18432\n\t"
        "s_add_u32 s60, %[rotX], 512\n\t"
        "s_and_b32 s60, s60, 4095\n\t"
        "buffer_load_dwordx4 %[vA0], %[srdX], s60 offen lds\n\t"
        "buffer_load_dwordx4 %[vA1], %[srdX], s60 offen offset:1024 lds\n\t"
        "s_add_u32 m0, %[ldsw], 10240\n\t"
        "s_add_u32 s61, %[rotW], 1048576\n\t"
        "s_and_b32 s61, s61, 0x7fffff\n\t"
        "buffer_load_dwordx4 %[vB0], %[srdW], s61 offen lds\n\t"
        "buffer_load_dwordx4 %[vB1], %[srdW], s61 offen offset:1024 lds\n\t"
        "buffer_load_dwordx4 %[vB2], %[srdW], s61 offen offset:2048 lds\n\t"
        "buffer_load_dwordx4 %[vB3], %[srdW], s61 offen offset:3072 lds\n\t"
        "s_add_u32 m0, %[ldsw], 14336\n\t"
        "s_add_u32 s60, s61, 131072\n\t"
        "buffer_load_dwordx4 %[vB0], %[srdW], s60 offen lds\n\t"
        "buffer_load_dwordx4 %[vB1], %[srdW], s60 offen offset:1024 lds\n\t"
        "buffer_load_dwordx4 %[vB2], %[srdW], s60 offen offset:2048 lds\n\t"
        "buffer_load_dwordx4 %[vB3], %[srdW], s60 offen offset:3072 lds\n\t"
        "s_add_u32 m0, %[ldsw], 28672\n\t"
        "s_add_u32 s61, %[rotX], 1024\n\t"
        "s_and_b32 s61, s61, 4095\n\t"
        "buffer_load_dwordx4 %[vA0], %[srdX], s61 offen lds\n\t"
        "buffer_load_dwordx4 %[vA1], %[srdX], s61 offen offset:1024 lds\n\t"
        "s_add_u32 m0, %[ldsw], 20480\n\t"
        "s_add_u32 s60, %[rotW], 2097152\n\t"
        "s_and_b32 s60, s60, 0x7fffff\n\t"
        "buffer_load_dwordx4 %[vB0], %[srdW], s60 offen lds\n\t"
        "buffer_load_dwordx4 %[vB1], %[srdW], s60 offen offset:1024 lds\n\t"
        "buffer_load_dwordx4 %[vB2], %[srdW], s60 offen offset:2048 lds\n\t"
        "buffer_load_dwordx4 %[vB3], %[srdW], s60 offen offset:3072 lds\n\t"
        "s_add_u32 m0, %[ldsw], 24576\n\t"
        "s_add_u32 s61, s60, 131072\n\t"
        "buffer_load_dwordx4 %[vB0], %[srdW], s61 offen lds\n\t"
        "buffer_load_dwordx4 %[vB1], %[srdW], s61 offen offset:1024 lds\n\t"
        "buffer_load_dwordx4 %[vB2], %[srdW], s61 offen offset:2048 lds\n\t"
        "buffer_load_dwordx4 %[vB3], %[srdW], s61 offen offset:3072 lds\n\t"
        :
        : [vA0] "v"(vA0), [vA1] "v"(vA1), [vB0] "v"(vB0), [vB1] "v"(vB1), [vB2] "v"(vB2), [vB3] "v"(vB3),
          [srdX] "s"(srdX), [srdW] "s"(srdW), [ldsw] "s"(ldsw), [rotW] "s"(rotW), [rotX] "s"(rotX)
        : "s60", "s61", "scc", "memory");
    const unsigned rem = __builtin_amdgcn_readfirstlane(dest != (unsigned)p.rank ? 1u : 0u);
    const unsigned tick = __builtin_amdgcn_readfirstlane((wave == 0 || wave == 3) ? 1u : 0u);
    const char* cnt = p.myws + WS_CNT + bid * WS_LINE + (wave & 1) * 64;
    unsigned ticket, bad;
    const unsigned fr = lane & 15, fg = lane >> 4, jh = wave & 1;
    const unsigned rA0 = ldsw + fr * 128 + ((fg ^ ((lane >> 1) & 7)) * 16);
    const unsigned q = fr >> 2, pp = lane & 3;
    const unsigned fB = (q >> 1) | ((fg & 1) << 1);
    const unsigned rB0 = (ldsw + (8 * fg + q) * 128 + fB * 32 + (pp >> 1) * 16 + (pp & 1) * 8) ^ (jh * 64);
    const unsigned lds0 = (unsigned)(size_t)(LAS char*)smem;
    const unsigned vW = ldsw + lane * 16 + jh * 2048;
    const unsigned vR1 = lds0 + ((wave + 1) & 3) * WAVE_LDS + lane * 16 + jh * 2048;
    const unsigned vR2 = lds0 + ((wave + 2) & 3) * WAVE_LDS + lane * 16 + jh * 2048;
    const unsigned vR3 = lds0 + ((wave + 3) & 3) * WAVE_LDS + lane * 16 + jh * 2048;
    const unsigned vO = (p.rank * MLOC + m0 + fr) * (NLOC * 2) + nl0 * 2 + jh * 64 + fg * 16;

    char* myws = p.myws;
    const char* outb = (const char*)p.out[dest];
    const char* flagp = (const char*)p.ws[dest] + WS_FLAG + (p.rank * 32 + tile) * WS_LINE;
    const char* inp = myws + WS_FLAG + (dest * 32 + tile) * WS_LINE;
    const char* tmo = myws + WS_TMO;
    asm volatile(
        "s_cmp_eq_u32 %[tick], 0\n\t"
        "s_cbranch_scc1 1f\n\t"
        "v_mov_b32 v211, 0\n\t"
        "v_mov_b32 v212, 1\n\t"
        "s_mov_b64 exec, 1\n\t"
        "global_atomic_add %[call], v211, v212, %[cnt] sc0\n\t"
        "s_mov_b64 exec, -1\n\t"
        "1:\n\t"
        "v_xor_b32 v204, 64, %[rA0]\n\t"
        "v_xor_b32 v205, 32, %[rB0]\n\t"
        "v_xor_b32 v206, 64, %[rB0]\n\t"
        "v_xor_b32 v207, 96, %[rB0]\n\t"
        "v_xor_b32 v208, 2048, %[vW]\n\t"
        "v_add_u32 v209, 8192, %[vO]\n\t"
        "s_add_u32 m0, %[ldsw], 38912\n\t"
        "s_add_u32 s60, %[rotX], 1536\n\t"
        "s_and_b32 s60, s60, 4095\n\t"
        "buffer_load_dwordx4 %[vA0], %[srdX], s60 offen lds\n\t"
        "buffer_load_dwordx4 %[vA1], %[srdX], s60 offen offset:1024 lds\n\t"
        "s_add_u32 m0, %[ldsw], 30720\n\t"
        "s_add_u32 s61, %[rotW], 3145728\n\t"
        "s_and_b32 s61, s61, 0x7fffff\n\t"
        "buffer_load_dwordx4 %[vB0], %[srdW], s61 offen lds\n\t"
        "s_waitcnt vmcnt(23)\n\t"
        "ds_read_b128 v[64:67], %[rA0] offset:8192\n\t"
        "ds_read_b64_tr_b16 a[0:1], %[rB0] offset:0\n\t"
        "ds_read_b64_tr_b16 a[2:3], %[rB0] offset:512\n\t"
        "ds_read_b64_tr_b16 a[4:5], v205 offset:0\n\t"
        "buffer_load_dwordx4 %[vB1], %[srdW], s61 offen offset:1024 lds\n\t"
        "ds_read_b64_tr_b16 a[6:7], v205 offset:512\n\t"
        "ds_read_b64_tr_b16 a[8:9], v206 offset:0\n\t"
        "ds_read_b64_tr_b16 a[10:11], v206 offset:512\n\t"
        "ds_read_b64_tr_b16 a[12:13], v207 offset:0\n\t"
        "buffer_load_dwordx4 %[vB2], %[srdW], s61 offen offset:2048 lds\n\t"
        "ds_read_b64_tr_b16 a[14:15], v207 offset:512\n\t"
        "ds_read_b128 v[68:71], v204 offset:8192\n\t"
        "ds_read_b64_tr_b16 a[16:17], %[rB0] offset:4096\n\t"
        "ds_read_b64_tr_b16 a[18:19], %[rB0] offset:4608\n\t"
        "buffer_load_dwordx4 %[vB3], %[srdW], s61 offen offset:3072 lds\n\t"
        "ds_read_b64_tr_b16 a[20:21], v205 offset:4096\n\t"
        "ds_read_b64_tr_b16 a[22:23], v205 offset:4608\n\t"
        "ds_read_b64_tr_b16 a[24:25], v206 offset:4096\n\t"
        "ds_read_b64_tr_b16 a[26:27], v206 offset:4608\n\t"
        "s_add_u32 m0, %[ldsw], 34816\n\t"
        "s_add_u32 s60, s61, 131072\n\t"
        "buffer_load_dwordx4 %[vB0], %[srdW], s60 offen lds\n\t"
        "ds_read_b64_tr_b16 a[28:29], v207 offset:4096\n\t"
        "ds_read_b64_tr_b16 a[30:31], v207 offset:4608\n\t"
        "buffer_load_dwordx4 %[vB1], %[srdW], s60 offen offset:1024 lds\n\t"
        "buffer_load_dwordx4 %[vB2], %[srdW], s60 offen offset:2048 lds\n\t"
        "buffer_load_dwordx4 %[vB3], %[srdW], s60 offen offset:3072 lds\n\t"
        "s_waitcnt lgkmcnt(0)\n\t"
        "s_add_u32 m0, %[ldsw], 8192\n\t"
        "s_add_u32 s61, %[rotX], 2048\n\t"
        "s_and_b32 s61, s61, 4095\n\t"
        "buffer_load_dwordx4 %[vA0], %[srdX], s61 offen lds\n\t"
        "v_mfma_f32_16x16x32_bf16 v[80:83], a[0:3], v[64:67], 0\n\t"
        "v_mfma_f32_16x16x32_bf16 v[84:87], a[4:7], v[64:67], 0\n\t"
        "v_mfma_f32_16x16x32_bf16 v[88:91], a[8:11], v[64:67], 0\n\t"
        "buffer_load_dwordx4 %[vA1], %[srdX], s61 offen offset:1024 lds\n\t"
        "v_mfma_f32_16x16x32_bf16 v[92:95], a[12:15], v[64:67], 0\n\t"
        "v_mfma_f32_16x16x32_bf16 v[80:83], a[16:19], v[68:71], v[80:83]\n\t"
        "v_mfma_f32_16x16x32_bf16 v[84:87], a[20:23], v[68:71], v[84:87]\n\t"
        "s_add_u32 m0, %[ldsw], 0\n\t"
        "s_add_u32 s60, %[rotW], 4194304\n\t"
        "s_and_b32 s60, s60, 0x7fffff\n\t"
        "buffer_load_dwordx4 %[vB0], %[srdW], s60 offen lds\n\t"
        "v_mfma_f32_16x16x32_bf16 v[88:91], a[24:27], v[68:71], v[88:91]\n\t"
        "v_mfma_f32_16x16x32_bf16 v[92:95], a[28:31], v[68:71], v[92:95]\n\t"
        "s_waitcnt vmcnt(23)\n\t"
        "ds_read_b128 v[72:75], %[rA0] offset:18432\n\t"
        "ds_read_b64_tr_b16 a[32:33], %[rB0] offset:10240\n\t"
        "ds_read_b64_tr_b16 a[34:35], %[rB0] offset:10752\n\t"
        "ds_read_b64_tr_b16 a[36:37], v205 offset:10240\n\t"
        "buffer_load_dwordx4 %[vB1], %[srdW], s60 offen offset:1024 lds\n\t"
        "ds_read_b64_tr_b16 a[38:39], v205 offset:10752\n\t"
        "ds_read_b64_tr_b16 a[40:41], v206 offset:10240\n\t"
        "ds_read_b64_tr_b16 a[42:43], v206 offset:10752\n\t"
        "ds_read_b64_tr_b16 a[44:45], v207 offset:10240\n\t"
        "buffer_load_dwordx4 %[vB2], %[srdW], s60 offen offset:2048 lds\n\t"
        "ds_read_b64_tr_b16 a[46:47], v207 offset:10752\n\t"
        "ds_read_b128 v[76:79], v204 offset:18432\n\t"
        "ds_read_b64_tr_b16 a[48:49], %[rB0] offset:14336\n\t"
        "ds_read_b64_tr_b16 a[50:51], %[rB0] offset:14848\n\t"
        "buffer_load_dwordx4 %[vB3], %[srdW], s60 offen offset:3072 lds\n\t"
        "ds_read_b64_tr_b16 a[52:53], v205 offset:14336\n\t"
        "ds_read_b64_tr_b16 a[54:55], v205 offset:14848\n\t"
        "ds_read_b64_tr_b16 a[56:57], v206 offset:14336\n\t"
        "ds_read_b64_tr_b16 a[58:59], v206 offset:14848\n\t"
        "s_add_u32 m0, %[ldsw], 4096\n\t"
        "s_add_u32 s61, s60, 131072\n\t"
        "buffer_load_dwordx4 %[vB0], %[srdW], s61 offen lds\n\t"
        "ds_read_b64_tr_b16 a[60:61], v207 offset:14336\n\t"
        "ds_read_b64_tr_b16 a[62:63], v207 offset:14848\n\t"
        "buffer_load_dwordx4 %[vB1], %[srdW], s61 offen offset:1024 lds\n\t"
        "buffer_load_dwordx4 %[vB2], %[srdW], s61 offen offset:2048 lds\n\t"
        "buffer_load_dwordx4 %[vB3], %[srdW], s61 offen offset:3072 lds\n\t"
        "s_waitcnt lgkmcnt(0)\n\t"
        "s_add_u32 m0, %[ldsw], 18432\n\t"
        "s_add_u32 s60, %[rotX], 2560\n\t"
        "s_and_b32 s60, s60, 4095\n\t"
        "buffer_load_dwordx4 %[vA0], %[srdX], s60 offen lds\n\t"
        "v_mfma_f32_16x16x32_bf16 v[80:83], a[32:35], v[72:75], v[80:83]\n\t"
        "v_mfma_f32_16x16x32_bf16 v[84:87], a[36:39], v[72:75], v[84:87]\n\t"
        "v_mfma_f32_16x16x32_bf16 v[88:91], a[40:43], v[72:75], v[88:91]\n\t"
        "buffer_load_dwordx4 %[vA1], %[srdX], s60 offen offset:1024 lds\n\t"
        "v_mfma_f32_16x16x32_bf16 v[92:95], a[44:47], v[72:75], v[92:95]\n\t"
        "v_mfma_f32_16x16x32_bf16 v[80:83], a[48:51], v[76:79], v[80:83]\n\t"
        "v_mfma_f32_16x16x32_bf16 v[84:87], a[52:55], v[76:79], v[84:87]\n\t"
        "s_add_u32 m0, %[ldsw], 10240\n\t"
        "s_add_u32 s61, %[rotW], 5242880\n\t"
        "s_and_b32 s61, s61, 0x7fffff\n\t"
        "buffer_load_dwordx4 %[vB0], %[srdW], s61 offen lds\n\t"
        "v_mfma_f32_16x16x32_bf16 v[88:91], a[56:59], v[76:79], v[88:91]\n\t"
        "v_mfma_f32_16x16x32_bf16 v[92:95], a[60:63], v[76:79], v[92:95]\n\t"
        "s_waitcnt vmcnt(23)\n\t"
        "ds_read_b128 v[64:67], %[rA0] offset:28672\n\t"
        "ds_read_b64_tr_b16 a[64:65], %[rB0] offset:20480\n\t"
        "ds_read_b64_tr_b16 a[66:67], %[rB0] offset:20992\n\t"
        "ds_read_b64_tr_b16 a[68:69], v205 offset:20480\n\t"
        "buffer_load_dwordx4 %[vB1], %[srdW], s61 offen offset:1024 lds\n\t"
        "ds_read_b64_tr_b16 a[70:71], v205 offset:20992\n\t"
        "ds_read_b64_tr_b16 a[72:73], v206 offset:20480\n\t"
        "ds_read_b64_tr_b16 a[74:75], v206 offset:20992\n\t"
        "ds_read_b64_tr_b16 a[76:77], v207 offset:20480\n\t"
        "buffer_load_dwordx4 %[vB2], %[srdW], s61 offen offset:2048 lds\n\t"
        "ds_read_b64_tr_b16 a[78:79], v207 offset:20992\n\t"
        "ds_read_b128 v[68:71], v204 offset:28672\n\t"
        "ds_read_b64_tr_b16 a[80:81], %[rB0] offset:24576\n\t"
        "ds_read_b64_tr_b16 a[82:83], %[rB0] offset:25088\n\t"
        "buffer_load_dwordx4 %[vB3], %[srdW], s61 offen offset:3072 lds\n\t"
        "ds_read_b64_tr_b16 a[84:85], v205 offset:24576\n\t"
        "ds_read_b64_tr_b16 a[86:87], v205 offset:25088\n\t"
        "ds_read_b64_tr_b16 a[88:89], v206 offset:24576\n\t"
        "ds_read_b64_tr_b16 a[90:91], v206 offset:25088\n\t"
        "s_add_u32 m0, %[ldsw], 14336\n\t"
        "s_add_u32 s60, s61, 131072\n\t"
        "buffer_load_dwordx4 %[vB0], %[srdW], s60 offen lds\n\t"
        "ds_read_b64_tr_b16 a[92:93], v207 offset:24576\n\t"
        "ds_read_b64_tr_b16 a[94:95], v207 offset:25088\n\t"
        "buffer_load_dwordx4 %[vB1], %[srdW], s60 offen offset:1024 lds\n\t"
        "buffer_load_dwordx4 %[vB2], %[srdW], s60 offen offset:2048 lds\n\t"
        "buffer_load_dwordx4 %[vB3], %[srdW], s60 offen offset:3072 lds\n\t"
        "s_waitcnt lgkmcnt(0)\n\t"
        "s_add_u32 m0, %[ldsw], 28672\n\t"
        "s_add_u32 s61, %[rotX], 3072\n\t"
        "s_and_b32 s61, s61, 4095\n\t"
        "buffer_load_dwordx4 %[vA0], %[srdX], s61 offen lds\n\t"
        "v_mfma_f32_16x16x32_bf16 v[80:83], a[64:67], v[64:67], v[80:83]\n\t"
        "v_mfma_f32_16x16x32_bf16 v[84:87], a[68:71], v[64:67], v[84:87]\n\t"
        "v_mfma_f32_16x16x32_bf16 v[88:91], a[72:75], v[64:67], v[88:91]\n\t"
        "buffer_load_dwordx4 %[vA1], %[srdX], s61 offen offset:1024 lds\n\t"
        "v_mfma_f32_16x16x32_bf16 v[92:95], a[76:79], v[64:67], v[92:95]\n\t"
        "v_mfma_f32_16x16x32_bf16 v[80:83], a[80:83], v[68:71], v[80:83]\n\t"
        "v_mfma_f32_16x16x32_bf16 v[84:87], a[84:87], v[68:71], v[84:87]\n\t"
        "s_add_u32 m0, %[ldsw], 20480\n\t"
        "s_add_u32 s60, %[rotW], 6291456\n\t"
        "s_and_b32 s60, s60, 0x7fffff\n\t"
        "buffer_load_dwordx4 %[vB0], %[srdW], s60 offen lds\n\t"
        "v_mfma_f32_16x16x32_bf16 v[88:91], a[88:91], v[68:71], v[88:91]\n\t"
        "v_mfma_f32_16x16x32_bf16 v[92:95], a[92:95], v[68:71], v[92:95]\n\t"
        "s_waitcnt vmcnt(23)\n\t"
        "ds_read_b128 v[72:75], %[rA0] offset:38912\n\t"
        "ds_read_b64_tr_b16 a[96:97], %[rB0] offset:30720\n\t"
        "ds_read_b64_tr_b16 a[98:99], %[rB0] offset:31232\n\t"
        "ds_read_b64_tr_b16 a[100:101], v205 offset:30720\n\t"
        "buffer_load_dwordx4 %[vB1], %[srdW], s60 offen offset:1024 lds\n\t"
        "ds_read_b64_tr_b16 a[102:103], v205 offset:31232\n\t"
        "ds_read_b64_tr_b16 a[104:105], v206 offset:30720\n\t"
        "ds_read_b64_tr_b16 a[106:107], v206 offset:31232\n\t"
        "ds_read_b64_tr_b16 a[108:109], v207 offset:30720\n\t"
        "buffer_load_dwordx4 %[vB2], %[srdW], s60 offen offset:2048 lds\n\t"
        "ds_read_b64_tr_b16 a[110:111], v207 offset:31232\n\t"
        "ds_read_b128 v[76:79], v204 offset:38912\n\t"
        "ds_read_b64_tr_b16 a[112:113], %[rB0] offset:34816\n\t"
        "ds_read_b64_tr_b16 a[114:115], %[rB0] offset:35328\n\t"
        "buffer_load_dwordx4 %[vB3], %[srdW], s60 offen offset:3072 lds\n\t"
        "ds_read_b64_tr_b16 a[116:117], v205 offset:34816\n\t"
        "ds_read_b64_tr_b16 a[118:119], v205 offset:35328\n\t"
        "ds_read_b64_tr_b16 a[120:121], v206 offset:34816\n\t"
        "ds_read_b64_tr_b16 a[122:123], v206 offset:35328\n\t"
        "s_add_u32 m0, %[ldsw], 24576\n\t"
        "s_add_u32 s61, s60, 131072\n\t"
        "buffer_load_dwordx4 %[vB0], %[srdW], s61 offen lds\n\t"
        "ds_read_b64_tr_b16 a[124:125], v207 offset:34816\n\t"
        "ds_read_b64_tr_b16 a[126:127], v207 offset:35328\n\t"
        "buffer_load_dwordx4 %[vB1], %[srdW], s61 offen offset:1024 lds\n\t"
        "buffer_load_dwordx4 %[vB2], %[srdW], s61 offen offset:2048 lds\n\t"
        "buffer_load_dwordx4 %[vB3], %[srdW], s61 offen offset:3072 lds\n\t"
        "s_waitcnt lgkmcnt(0)\n\t"
        "s_add_u32 m0, %[ldsw], 38912\n\t"
        "s_add_u32 s60, %[rotX], 3584\n\t"
        "s_and_b32 s60, s60, 4095\n\t"
        "buffer_load_dwordx4 %[vA0], %[srdX], s60 offen lds\n\t"
        "v_mfma_f32_16x16x32_bf16 v[80:83], a[96:99], v[72:75], v[80:83]\n\t"
        "v_mfma_f32_16x16x32_bf16 v[84:87], a[100:103], v[72:75], v[84:87]\n\t"
        "v_mfma_f32_16x16x32_bf16 v[88:91], a[104:107], v[72:75], v[88:91]\n\t"
        "buffer_load_dwordx4 %[vA1], %[srdX], s60 offen offset:1024 lds\n\t"
        "v_mfma_f32_16x16x32_bf16 v[92:95], a[108:111], v[72:75], v[92:95]\n\t"
        "v_mfma_f32_16x16x32_bf16 v[80:83], a[112:115], v[76:79], v[80:83]\n\t"
        "v_mfma_f32_16x16x32_bf16 v[84:87], a[116:119], v[76:79], v[84:87]\n\t"
        "s_add_u32 m0, %[ldsw], 30720\n\t"
        "s_add_u32 s61, %[rotW], 7340032\n\t"
        "s_and_b32 s61, s61, 0x7fffff\n\t"
        "buffer_load_dwordx4 %[vB0], %[srdW], s61 offen lds\n\t"
        "v_mfma_f32_16x16x32_bf16 v[88:91], a[120:123], v[76:79], v[88:91]\n\t"
        "v_mfma_f32_16x16x32_bf16 v[92:95], a[124:127], v[76:79], v[92:95]\n\t"
        "s_waitcnt vmcnt(23)\n\t"
        "ds_read_b128 v[64:67], %[rA0] offset:8192\n\t"
        "ds_read_b64_tr_b16 a[128:129], %[rB0] offset:0\n\t"
        "ds_read_b64_tr_b16 a[130:131], %[rB0] offset:512\n\t"
        "ds_read_b64_tr_b16 a[132:133], v205 offset:0\n\t"
        "buffer_load_dwordx4 %[vB1], %[srdW], s61 offen offset:1024 lds\n\t"
        "ds_read_b64_tr_b16 a[134:135], v205 offset:512\n\t"
        "ds_read_b64_tr_b16 a[136:137], v206 offset:0\n\t"
        "ds_read_b64_tr_b16 a[138:139], v206 offset:512\n\t"
        "ds_read_b64_tr_b16 a[140:141], v207 offset:0\n\t"
        "buffer_load_dwordx4 %[vB2], %[srdW], s61 offen offset:2048 lds\n\t"
        "ds_read_b64_tr_b16 a[142:143], v207 offset:512\n\t"
        "ds_read_b128 v[68:71], v204 offset:8192\n\t"
        "ds_read_b64_tr_b16 a[144:145], %[rB0] offset:4096\n\t"
        "ds_read_b64_tr_b16 a[146:147], %[rB0] offset:4608\n\t"
        "buffer_load_dwordx4 %[vB3], %[srdW], s61 offen offset:3072 lds\n\t"
        "ds_read_b64_tr_b16 a[148:149], v205 offset:4096\n\t"
        "ds_read_b64_tr_b16 a[150:151], v205 offset:4608\n\t"
        "ds_read_b64_tr_b16 a[152:153], v206 offset:4096\n\t"
        "ds_read_b64_tr_b16 a[154:155], v206 offset:4608\n\t"
        "s_add_u32 m0, %[ldsw], 34816\n\t"
        "s_add_u32 s60, s61, 131072\n\t"
        "buffer_load_dwordx4 %[vB0], %[srdW], s60 offen lds\n\t"
        "ds_read_b64_tr_b16 a[156:157], v207 offset:4096\n\t"
        "ds_read_b64_tr_b16 a[158:159], v207 offset:4608\n\t"
        "buffer_load_dwordx4 %[vB1], %[srdW], s60 offen offset:1024 lds\n\t"
        "buffer_load_dwordx4 %[vB2], %[srdW], s60 offen offset:2048 lds\n\t"
        "buffer_load_dwordx4 %[vB3], %[srdW], s60 offen offset:3072 lds\n\t"
        "s_cmp_gt_u32 %[wave], 1\n\t"
        "s_cbranch_scc1 20f\n\t"
        "s_waitcnt lgkmcnt(0)\n\t"
        "v_mfma_f32_16x16x32_bf16 v[80:83], a[128:131], v[64:67], v[80:83]\n\t"
        "v_mfma_f32_16x16x32_bf16 v[84:87], a[132:135], v[64:67], v[84:87]\n\t"
        "v_mfma_f32_16x16x32_bf16 v[88:91], a[136:139], v[64:67], v[88:91]\n\t"
        "v_mfma_f32_16x16x32_bf16 v[92:95], a[140:143], v[64:67], v[92:95]\n\t"
        "v_mfma_f32_16x16x32_bf16 v[80:83], a[144:147], v[68:71], v[80:83]\n\t"
        "v_mfma_f32_16x16x32_bf16 v[84:87], a[148:151], v[68:71], v[84:87]\n\t"
        "v_mfma_f32_16x16x32_bf16 v[88:91], a[152:155], v[68:71], v[88:91]\n\t"
        "v_mfma_f32_16x16x32_bf16 v[92:95], a[156:159], v[68:71], v[92:95]\n\t"
        "s_waitcnt vmcnt(20)\n\t"
        "ds_read_b128 v[72:75], %[rA0] offset:18432\n\t"
        "ds_read_b64_tr_b16 a[160:161], %[rB0] offset:10240\n\t"
        "ds_read_b64_tr_b16 a[162:163], %[rB0] offset:10752\n\t"
        "ds_read_b64_tr_b16 a[164:165], v205 offset:10240\n\t"
        "ds_read_b64_tr_b16 a[166:167], v205 offset:10752\n\t"
        "ds_read_b64_tr_b16 a[168:169], v206 offset:10240\n\t"
        "ds_read_b64_tr_b16 a[170:171], v206 offset:10752\n\t"
        "ds_read_b64_tr_b16 a[172:173], v207 offset:10240\n\t"
        "ds_read_b64_tr_b16 a[174:175], v207 offset:10752\n\t"
        "ds_read_b128 v[76:79], v204 offset:18432\n\t"
        "ds_read_b64_tr_b16 a[176:177], %[rB0] offset:14336\n\t"
        "ds_read_b64_tr_b16 a[178:179], %[rB0] offset:14848\n\t"
        "ds_read_b64_tr_b16 a[180:181], v205 offset:14336\n\t"
        "ds_read_b64_tr_b16 a[182:183], v205 offset:14848\n\t"
        "ds_read_b64_tr_b16 a[184:185], v206 offset:14336\n\t"
        "ds_read_b64_tr_b16 a[186:187], v206 offset:14848\n\t"
        "ds_read_b64_tr_b16 a[188:189], v207 offset:14336\n\t"
        "ds_read_b64_tr_b16 a[190:191], v207 offset:14848\n\t"
        "s_add_u32 m0, %[ldsw], 0\n\t"
        "s_add_u32 s61, %[rotX], 0\n\t"
        "s_and_b32 s61, s61, 4095\n\t"
        "s_add_u32 s61, s61, 65536\n\t"
        "buffer_load_dwordx4 %[vA0], %[srdX], s61 offen lds\n\t"
        "buffer_load_dwordx4 %[vA1], %[srdX], s61 offen offset:1024 lds\n\t"
        "s_add_u32 m0, %[ldsw], 2048\n\t"
        "s_add_u32 s60, %[rotX], 512\n\t"
        "s_and_b32 s60, s60, 4095\n\t"
        "s_add_u32 s60, s60, 65536\n\t"
        "buffer_load_dwordx4 %[vA0], %[srdX], s60 offen lds\n\t"
        "buffer_load_dwordx4 %[vA1], %[srdX], s60 offen offset:1024 lds\n\t"
        "s_waitcnt lgkmcnt(0)\n\t"
        "v_mfma_f32_16x16x32_bf16 v[80:83], a[160:163], v[72:75], v[80:83]\n\t"
        "v_mfma_f32_16x16x32_bf16 v[84:87], a[164:167], v[72:75], v[84:87]\n\t"
        "v_mfma_f32_16x16x32_bf16 v[88:91], a[168:171], v[72:75], v[88:91]\n\t"
        "v_mfma_f32_16x16x32_bf16 v[92:95], a[172:175], v[72:75], v[92:95]\n\t"
        "v_mfma_f32_16x16x32_bf16 v[80:83], a[176:179], v[76:79], v[80:83]\n\t"
        "v_mfma_f32_16x16x32_bf16 v[84:87], a[180:183], v[76:79], v[84:87]\n\t"
        "v_mfma_f32_16x16x32_bf16 v[88:91], a[184:187], v[76:79], v[88:91]\n\t"
        "v_mfma_f32_16x16x32_bf16 v[92:95], a[188:191], v[76:79], v[92:95]\n\t"
        "s_waitcnt vmcnt(14)\n\t"
        "ds_read_b128 v[64:67], %[rA0] offset:28672\n\t"
        "ds_read_b64_tr_b16 a[192:193], %[rB0] offset:20480\n\t"
        "ds_read_b64_tr_b16 a[194:195], %[rB0] offset:20992\n\t"
        "ds_read_b64_tr_b16 a[196:197], v205 offset:20480\n\t"
        "ds_read_b64_tr_b16 a[198:199], v205 offset:20992\n\t"
        "ds_read_b64_tr_b16 a[200:201], v206 offset:20480\n\t"
        "ds_read_b64_tr_b16 a[202:203], v206 offset:20992\n\t"
        "ds_read_b64_tr_b16 a[204:205], v207 offset:20480\n\t"
        "ds_read_b64_tr_b16 a[206:207], v207 offset:20992\n\t"
        "ds_read_b128 v[68:71], v204 offset:28672\n\t"
        "ds_read_b64_tr_b16 a[208:209], %[rB0] offset:24576\n\t"
        "ds_read_b64_tr_b16 a[210:211], %[rB0] offset:25088\n\t"
        "ds_read_b64_tr_b16 a[212:213], v205 offset:24576\n\t"
        "ds_read_b64_tr_b16 a[214:215], v205 offset:25088\n\t"
        "ds_read_b64_tr_b16 a[216:217], v206 offset:24576\n\t"
        "ds_read_b64_tr_b16 a[218:219], v206 offset:25088\n\t"
        "ds_read_b64_tr_b16 a[220:221], v207 offset:24576\n\t"
        "ds_read_b64_tr_b16 a[222:223], v207 offset:25088\n\t"
        "s_add_u32 m0, %[ldsw], 4096\n\t"
        "s_add_u32 s61, %[rotX], 1024\n\t"
        "s_and_b32 s61, s61, 4095\n\t"
        "s_add_u32 s61, s61, 65536\n\t"
        "buffer_load_dwordx4 %[vA0], %[srdX], s61 offen lds\n\t"
        "buffer_load_dwordx4 %[vA1], %[srdX], s61 offen offset:1024 lds\n\t"
        "s_add_u32 m0, %[ldsw], 6144\n\t"
        "s_add_u32 s60, %[rotX], 1536\n\t"
        "s_and_b32 s60, s60, 4095\n\t"
        "s_add_u32 s60, s60, 65536\n\t"
        "buffer_load_dwordx4 %[vA0], %[srdX], s60 offen lds\n\t"
        "buffer_load_dwordx4 %[vA1], %[srdX], s60 offen offset:1024 lds\n\t"
        "s_waitcnt lgkmcnt(0)\n\t"
        "s_add_u32 m0, %[ldsw], 10240\n\t"
        "s_add_u32 s61, %[rotX], 2048\n\t"
        "s_and_b32 s61, s61, 4095\n\t"
        "s_add_u32 s61, s61, 65536\n\t"
        "buffer_load_dwordx4 %[vA0], %[srdX], s61 offen lds\n\t"
        "v_mfma_f32_16x16x32_bf16 v[80:83], a[192:195], v[64:67], v[80:83]\n\t"
        "v_mfma_f32_16x16x32_bf16 v[84:87], a[196:199], v[64:67], v[84:87]\n\t"
        "v_mfma_f32_16x16x32_bf16 v[88:91], a[200:203], v[64:67], v[88:91]\n\t"
        "buffer_load_dwordx4 %[vA1], %[srdX], s61 offen offset:1024 lds\n\t"
        "v_mfma_f32_16x16x32_bf16 v[92:95], a[204:207], v[64:67], v[92:95]\n\t"
        "v_mfma_f32_16x16x32_bf16 v[80:83], a[208:211], v[68:71], v[80:83]\n\t"
        "v_mfma_f32_16x16x32_bf16 v[84:87], a[212:215], v[68:71], v[84:87]\n\t"
        "s_add_u32 m0, %[ldsw], 12288\n\t"
        "s_add_u32 s60, %[rotX], 2560\n\t"
        "s_and_b32 s60, s60, 4095\n\t"
        "s_add_u32 s60, s60, 65536\n\t"
        "buffer_load_dwordx4 %[vA0], %[srdX], s60 offen lds\n\t"
        "v_mfma_f32_16x16x32_bf16 v[88:91], a[216:219], v[68:71], v[88:91]\n\t"
        "v_mfma_f32_16x16x32_bf16 v[92:95], a[220:223], v[68:71], v[92:95]\n\t"
        "buffer_load_dwordx4 %[vA1], %[srdX], s60 offen offset:1024 lds\n\t"
        "s_waitcnt vmcnt(12)\n\t"
        "ds_read_b128 v[72:75], %[rA0] offset:38912\n\t"
        "ds_read_b64_tr_b16 a[224:225], %[rB0] offset:30720\n\t"
        "ds_read_b64_tr_b16 a[226:227], %[rB0] offset:31232\n\t"
        "ds_read_b64_tr_b16 a[228:229], v205 offset:30720\n\t"
        "ds_read_b64_tr_b16 a[230:231], v205 offset:31232\n\t"
        "ds_read_b64_tr_b16 a[232:233], v206 offset:30720\n\t"
        "ds_read_b64_tr_b16 a[234:235], v206 offset:31232\n\t"
        "ds_read_b64_tr_b16 a[236:237], v207 offset:30720\n\t"
        "ds_read_b64_tr_b16 a[238:239], v207 offset:31232\n\t"
        "ds_read_b128 v[76:79], v204 offset:38912\n\t"
        "ds_read_b64_tr_b16 a[240:241], %[rB0] offset:34816\n\t"
        "ds_read_b64_tr_b16 a[242:243], %[rB0] offset:35328\n\t"
        "ds_read_b64_tr_b16 a[244:245], v205 offset:34816\n\t"
        "ds_read_b64_tr_b16 a[246:247], v205 offset:35328\n\t"
        "ds_read_b64_tr_b16 a[248:249], v206 offset:34816\n\t"
        "ds_read_b64_tr_b16 a[250:251], v206 offset:35328\n\t"
        "ds_read_b64_tr_b16 a[252:253], v207 offset:34816\n\t"
        "ds_read_b64_tr_b16 a[254:255], v207 offset:35328\n\t"
        "s_waitcnt lgkmcnt(9)\n\t"
        "v_mfma_f32_16x16x32_bf16 v[80:83], a[224:227], v[72:75], v[80:83]\n\t"
        "v_mfma_f32_16x16x32_bf16 v[84:87], a[228:231], v[72:75], v[84:87]\n\t"
        "v_mfma_f32_16x16x32_bf16 v[88:91], a[232:235], v[72:75], v[88:91]\n\t"
        "v_mfma_f32_16x16x32_bf16 v[92:95], a[236:239], v[72:75], v[92:95]\n\t"
        "s_waitcnt lgkmcnt(0)\n\t"
        "v_mfma_f32_16x16x32_bf16 v[80:83], a[240:243], v[76:79], v[80:83]\n\t"
        "v_mfma_f32_16x16x32_bf16 v[84:87], a[244:247], v[76:79], v[84:87]\n\t"
        "v_mfma_f32_16x16x32_bf16 v[88:91], a[248:251], v[76:79], v[88:91]\n\t"
        "v_mfma_f32_16x16x32_bf16 v[92:95], a[252:255], v[76:79], v[92:95]\n\t"
        "s_nop 7\n\t"
        "s_nop 1\n\t"
        "ds_write_b128 v208, v[88:91] offset:20480\n\t"
        "ds_write_b128 v208, v[92:95] offset:21504\n\t"
        "s_add_u32 m0, %[ldsw], 14336\n\t"
        "s_add_u32 s61, %[rotX], 3072\n\t"
        "s_and_b32 s61, s61, 4095\n\t"
        "s_add_u32 s61, s61, 65536\n\t"
        "buffer_load_dwordx4 %[vA0], %[srdX], s61 offen lds\n\t"
        "buffer_load_dwordx4 %[vA1], %[srdX], s61 offen offset:1024 lds\n\t"
        "s_waitcnt lgkmcnt(0)\n\t"
        "s_barrier\n\t"
        "ds_read_b128 v[176:179], %[vR1] offset:20480\n\t"
        "ds_read_b128 v[180:183], %[vR1] offset:21504\n\t"
        "ds_read_b128 v[184:187], %[vR2] offset:20480\n\t"
        "ds_read_b128 v[188:191], %[vR2] offset:21504\n\t"
        "ds_read_b128 v[192:195], %[vR3] offset:20480\n\t"
        "ds_read_b128 v[196:199], %[vR3] offset:21504\n\t"
        "s_add_u32 m0, %[ldsw], 16384\n\t"
        "s_add_u32 s60, %[rotX], 3584\n\t"
        "s_and_b32 s60, s60, 4095\n\t"
        "s_add_u32 s60, s60, 65536\n\t"
        "buffer_load_dwordx4 %[vA0], %[srdX], s60 offen lds\n\t"
        "buffer_load_dwordx4 %[vA1], %[srdX], s60 offen offset:1024 lds\n\t"
        "s_waitcnt vmcnt(8)\n\t"
        "ds_read_b128 v[112:115], %[rA0] offset:0\n\t"
        "ds_read_b128 v[116:119], v204 offset:0\n\t"
        "ds_read_b128 v[120:123], %[rA0] offset:2048\n\t"
        "ds_read_b128 v[124:127], v204 offset:2048\n\t"
        "ds_read_b128 v[128:131], %[rA0] offset:4096\n\t"
        "ds_read_b128 v[132:135], v204 offset:4096\n\t"
        "ds_read_b128 v[136:139], %[rA0] offset:6144\n\t"
        "ds_read_b128 v[140:143], v204 offset:6144\n\t"
        "s_waitcnt lgkmcnt(8)\n\t"
        "v_pk_add_f32 v[80:81], v[80:81], v[176:177]\n\t"
        "v_pk_add_f32 v[184:185], v[184:185], v[192:193]\n\t"
        "v_pk_add_f32 v[82:83], v[82:83], v[178:179]\n\t"
        "v_pk_add_f32 v[186:187], v[186:187], v[194:195]\n\t"
        "v_pk_add_f32 v[84:85], v[84:85], v[180:181]\n\t"
        "v_pk_add_f32 v[188:189], v[188:189], v[196:197]\n\t"
        "v_pk_add_f32 v[86:87], v[86:87], v[182:183]\n\t"
        "v_pk_add_f32 v[190:191], v[190:191], v[198:199]\n\t"
        "v_pk_add_f32 v[80:81], v[80:81], v[184:185]\n\t"
        "v_pk_add_f32 v[82:83], v[82:83], v[186:187]\n\t"
        "v_pk_add_f32 v[84:85], v[84:85], v[188:189]\n\t"
        "v_pk_add_f32 v[86:87], v[86:87], v[190:191]\n\t"
        "s_nop 0\n\t"
        "v_cvt_pk_bf16_f32 v200, v80, v81\n\t"
        "v_cvt_pk_bf16_f32 v201, v82, v83\n\t"
        "v_cvt_pk_bf16_f32 v202, v84, v85\n\t"
        "v_cvt_pk_bf16_f32 v203, v86, v87\n\t"
        "s_nop 1\n\t"
        "v_permlane32_swap_b32 v200, v202\n\t"
        "v_permlane32_swap_b32 v201, v203\n\t"
        "s_nop 0\n\t"
        "v_permlane16_swap_b32 v200, v202\n\t"
        "v_permlane16_swap_b32 v201, v203\n\t"
        "global_store_dwordx4 %[vO], v[200:203], %[outb] sc0 sc1\n\t"
        "s_nop 1\n\t"
        "s_waitcnt vmcnt(1)\n\t"
        "ds_read_b128 v[144:147], %[rA0] offset:10240\n\t"
        "ds_read_b128 v[148:151], v204 offset:10240\n\t"
        "ds_read_b128 v[152:155], %[rA0] offset:12288\n\t"
        "ds_read_b128 v[156:159], v204 offset:12288\n\t"
        "ds_read_b128 v[160:163], %[rA0] offset:14336\n\t"
        "ds_read_b128 v[164:167], v204 offset:14336\n\t"
        "ds_read_b128 v[168:171], %[rA0] offset:16384\n\t"
        "ds_read_b128 v[172:175], v204 offset:16384\n\t"
        "s_waitcnt lgkmcnt(14)\n\t"
        "v_mfma_f32_16x16x32_bf16 v[96:99], a[0:3], v[112:115], 0\n\t"
        "v_mfma_f32_16x16x32_bf16 v[100:103], a[4:7], v[112:115], 0\n\t"
        "v_mfma_f32_16x16x32_bf16 v[104:107], a[8:11], v[112:115], 0\n\t"
        "v_mfma_f32_16x16x32_bf16 v[108:111], a[12:15], v[112:115], 0\n\t"
        "v_mfma_f32_16x16x32_bf16 v[96:99], a[16:19], v[116:119], v[96:99]\n\t"
        "v_mfma_f32_16x16x32_bf16 v[100:103], a[20:23], v[116:119], v[100:103]\n\t"
        "v_mfma_f32_16x16x32_bf16 v[104:107], a[24:27], v[116:119], v[104:107]\n\t"
        "v_mfma_f32_16x16x32_bf16 v[108:111], a[28:31], v[116:119], v[108:111]\n\t"
        "s_waitcnt lgkmcnt(12)\n\t"
        "v_mfma_f32_16x16x32_bf16 v[96:99], a[32:35], v[120:123], v[96:99]\n\t"
        "v_mfma_f32_16x16x32_bf16 v[100:103], a[36:39], v[120:123], v[100:103]\n\t"
        "v_mfma_f32_16x16x32_bf16 v[104:107], a[40:43], v[120:123], v[104:107]\n\t"
        "v_mfma_f32_16x16x32_bf16 v[108:111], a[44:47], v[120:123], v[108:111]\n\t"
        "v_mfma_f32_16x16x32_bf16 v[96:99], a[48:51], v[124:127], v[96:99]\n\t"
        "v_mfma_f32_16x16x32_bf16 v[100:103], a[52:55], v[124:127], v[100:103]\n\t"
        "v_mfma_f32_16x16x32_bf16 v[104:107], a[56:59], v[124:127], v[104:107]\n\t"
        "v_mfma_f32_16x16x32_bf16 v[108:111], a[60:63], v[124:127], v[108:111]\n\t"
        "s_waitcnt lgkmcnt(10)\n\t"
        "v_mfma_f32_16x16x32_bf16 v[96:99], a[64:67], v[128:131], v[96:99]\n\t"
        "v_mfma_f32_16x16x32_bf16 v[100:103], a[68:71], v[128:131], v[100:103]\n\t"
        "v_mfma_f32_16x16x32_bf16 v[104:107], a[72:75], v[128:131], v[104:107]\n\t"
        "v_mfma_f32_16x16x32_bf16 v[108:111], a[76:79], v[128:131], v[108:111]\n\t"
        "v_mfma_f32_16x16x32_bf16 v[96:99], a[80:83], v[132:135], v[96:99]\n\t"
        "v_mfma_f32_16x16x32_bf16 v[100:103], a[84:87], v[132:135], v[100:103]\n\t"
        "v_mfma_f32_16x16x32_bf16 v[104:107], a[88:91], v[132:135], v[104:107]\n\t"
        "v_mfma_f32_16x16x32_bf16 v[108:111], a[92:95], v[132:135], v[108:111]\n\t"
        "s_waitcnt lgkmcnt(8)\n\t"
        "v_mfma_f32_16x16x32_bf16 v[96:99], a[96:99], v[136:139], v[96:99]\n\t"
        "v_mfma_f32_16x16x32_bf16 v[100:103], a[100:103], v[136:139], v[100:103]\n\t"
        "v_mfma_f32_16x16x32_bf16 v[104:107], a[104:107], v[136:139], v[104:107]\n\t"
        "v_mfma_f32_16x16x32_bf16 v[108:111], a[108:111], v[136:139], v[108:111]\n\t"
        "v_mfma_f32_16x16x32_bf16 v[96:99], a[112:115], v[140:143], v[96:99]\n\t"
        "v_mfma_f32_16x16x32_bf16 v[100:103], a[116:119], v[140:143], v[100:103]\n\t"
        "v_mfma_f32_16x16x32_bf16 v[104:107], a[120:123], v[140:143], v[104:107]\n\t"
        "v_mfma_f32_16x16x32_bf16 v[108:111], a[124:127], v[140:143], v[108:111]\n\t"
        "s_waitcnt lgkmcnt(6)\n\t"
        "v_mfma_f32_16x16x32_bf16 v[96:99], a[128:131], v[144:147], v[96:99]\n\t"
        "v_mfma_f32_16x16x32_bf16 v[100:103], a[132:135], v[144:147], v[100:103]\n\t"
        "v_mfma_f32_16x16x32_bf16 v[104:107], a[136:139], v[144:147], v[104:107]\n\t"
        "v_mfma_f32_16x16x32_bf16 v[108:111], a[140:143], v[144:147], v[108:111]\n\t"
        "v_mfma_f32_16x16x32_bf16 v[96:99], a[144:147], v[148:151], v[96:99]\n\t"
        "v_mfma_f32_16x16x32_bf16 v[100:103], a[148:151], v[148:151], v[100:103]\n\t"
        "v_mfma_f32_16x16x32_bf16 v[104:107], a[152:155], v[148:151], v[104:107]\n\t"
        "v_mfma_f32_16x16x32_bf16 v[108:111], a[156:159], v[148:151], v[108:111]\n\t"
        "s_waitcnt lgkmcnt(4)\n\t"
        "v_mfma_f32_16x16x32_bf16 v[96:99], a[160:163], v[152:155], v[96:99]\n\t"
        "v_mfma_f32_16x16x32_bf16 v[100:103], a[164:167], v[152:155], v[100:103]\n\t"
        "v_mfma_f32_16x16x32_bf16 v[104:107], a[168:171], v[152:155], v[104:107]\n\t"
        "v_mfma_f32_16x16x32_bf16 v[108:111], a[172:175], v[152:155], v[108:111]\n\t"
        "v_mfma_f32_16x16x32_bf16 v[96:99], a[176:179], v[156:159], v[96:99]\n\t"
        "v_mfma_f32_16x16x32_bf16 v[100:103], a[180:183], v[156:159], v[100:103]\n\t"
        "v_mfma_f32_16x16x32_bf16 v[104:107], a[184:187], v[156:159], v[104:107]\n\t"
        "v_mfma_f32_16x16x32_bf16 v[108:111], a[188:191], v[156:159], v[108:111]\n\t"
        "s_waitcnt lgkmcnt(2)\n\t"
        "v_mfma_f32_16x16x32_bf16 v[96:99], a[192:195], v[160:163], v[96:99]\n\t"
        "v_mfma_f32_16x16x32_bf16 v[100:103], a[196:199], v[160:163], v[100:103]\n\t"
        "v_mfma_f32_16x16x32_bf16 v[104:107], a[200:203], v[160:163], v[104:107]\n\t"
        "v_mfma_f32_16x16x32_bf16 v[108:111], a[204:207], v[160:163], v[108:111]\n\t"
        "v_mfma_f32_16x16x32_bf16 v[96:99], a[208:211], v[164:167], v[96:99]\n\t"
        "v_mfma_f32_16x16x32_bf16 v[100:103], a[212:215], v[164:167], v[100:103]\n\t"
        "v_mfma_f32_16x16x32_bf16 v[104:107], a[216:219], v[164:167], v[104:107]\n\t"
        "v_mfma_f32_16x16x32_bf16 v[108:111], a[220:223], v[164:167], v[108:111]\n\t"
        "s_waitcnt lgkmcnt(0)\n\t"
        "v_mfma_f32_16x16x32_bf16 v[96:99], a[224:227], v[168:171], v[96:99]\n\t"
        "v_mfma_f32_16x16x32_bf16 v[100:103], a[228:231], v[168:171], v[100:103]\n\t"
        "v_mfma_f32_16x16x32_bf16 v[104:107], a[232:235], v[168:171], v[104:107]\n\t"
        "v_mfma_f32_16x16x32_bf16 v[108:111], a[236:239], v[168:171], v[108:111]\n\t"
        "v_mfma_f32_16x16x32_bf16 v[96:99], a[240:243], v[172:175], v[96:99]\n\t"
        "v_mfma_f32_16x16x32_bf16 v[100:103], a[244:247], v[172:175], v[100:103]\n\t"
        "v_mfma_f32_16x16x32_bf16 v[104:107], a[248:251], v[172:175], v[104:107]\n\t"
        "v_mfma_f32_16x16x32_bf16 v[108:111], a[252:255], v[172:175], v[108:111]\n\t"
        "s_nop 7\n\t"
        "s_nop 1\n\t"
        "ds_write_b128 %[vW], v[96:99] offset:30720\n\t"
        "ds_write_b128 %[vW], v[100:103] offset:31744\n\t"
        "ds_write_b128 v208, v[104:107] offset:30720\n\t"
        "ds_write_b128 v208, v[108:111] offset:31744\n\t"
        "s_waitcnt lgkmcnt(0)\n\t"
        "s_barrier\n\t"
        "s_waitcnt vmcnt(0)\n\t"
        "s_branch 21f\n\t"
        "20:\n\t"
        "s_waitcnt lgkmcnt(0)\n\t"
        "v_mfma_f32_16x16x32_bf16 v[80:83], a[128:131], v[64:67], v[80:83]\n\t"
        "v_mfma_f32_16x16x32_bf16 v[84:87], a[132:135], v[64:67], v[84:87]\n\t"
        "v_mfma_f32_16x16x32_bf16 v[88:91], a[136:139], v[64:67], v[88:91]\n\t"
        "v_mfma_f32_16x16x32_bf16 v[92:95], a[140:143], v[64:67], v[92:95]\n\t"
        "v_mfma_f32_16x16x32_bf16 v[80:83], a[144:147], v[68:71], v[80:83]\n\t"
        "v_mfma_f32_16x16x32_bf16 v[84:87], a[148:151], v[68:71], v[84:87]\n\t"
        "v_mfma_f32_16x16x32_bf16 v[88:91], a[152:155], v[68:71], v[88:91]\n\t"
        "v_mfma_f32_16x16x32_bf16 v[92:95], a[156:159], v[68:71], v[92:95]\n\t"
        "s_waitcnt vmcnt(20)\n\t"
        "ds_read_b128 v[72:75], %[rA0] offset:18432\n\t"
        "ds_read_b64_tr_b16 a[160:161], %[rB0] offset:10240\n\t"
        "ds_read_b64_tr_b16 a[162:163], %[rB0] offset:10752\n\t"
        "ds_read_b64_tr_b16 a[164:165], v205 offset:10240\n\t"
        "ds_read_b64_tr_b16 a[166:167], v205 offset:10752\n\t"
        "ds_read_b64_tr_b16 a[168:169], v206 offset:10240\n\t"
        "ds_read_b64_tr_b16 a[170:171], v206 offset:10752\n\t"
        "ds_read_b64_tr_b16 a[172:173], v207 offset:10240\n\t"
        "ds_read_b64_tr_b16 a[174:175], v207 offset:10752\n\t"
        "ds_read_b128 v[76:79], v204 offset:18432\n\t"
        "ds_read_b64_tr_b16 a[176:177], %[rB0] offset:14336\n\t"
        "ds_read_b64_tr_b16 a[178:179], %[rB0] offset:14848\n\t"
        "ds_read_b64_tr_b16 a[180:181], v205 offset:14336\n\t"
        "ds_read_b64_tr_b16 a[182:183], v205 offset:14848\n\t"
        "ds_read_b64_tr_b16 a[184:185], v206 offset:14336\n\t"
        "ds_read_b64_tr_b16 a[186:187], v206 offset:14848\n\t"
        "ds_read_b64_tr_b16 a[188:189], v207 offset:14336\n\t"
        "ds_read_b64_tr_b16 a[190:191], v207 offset:14848\n\t"
        "s_waitcnt lgkmcnt(0)\n\t"
        "v_mfma_f32_16x16x32_bf16 v[80:83], a[160:163], v[72:75], v[80:83]\n\t"
        "v_mfma_f32_16x16x32_bf16 v[84:87], a[164:167], v[72:75], v[84:87]\n\t"
        "v_mfma_f32_16x16x32_bf16 v[88:91], a[168:171], v[72:75], v[88:91]\n\t"
        "v_mfma_f32_16x16x32_bf16 v[92:95], a[172:175], v[72:75], v[92:95]\n\t"
        "v_mfma_f32_16x16x32_bf16 v[80:83], a[176:179], v[76:79], v[80:83]\n\t"
        "v_mfma_f32_16x16x32_bf16 v[84:87], a[180:183], v[76:79], v[84:87]\n\t"
        "v_mfma_f32_16x16x32_bf16 v[88:91], a[184:187], v[76:79], v[88:91]\n\t"
        "v_mfma_f32_16x16x32_bf16 v[92:95], a[188:191], v[76:79], v[92:95]\n\t"
        "s_waitcnt vmcnt(10)\n\t"
        "ds_read_b128 v[64:67], %[rA0] offset:28672\n\t"
        "ds_read_b64_tr_b16 a[192:193], %[rB0] offset:20480\n\t"
        "ds_read_b64_tr_b16 a[194:195], %[rB0] offset:20992\n\t"
        "ds_read_b64_tr_b16 a[196:197], v205 offset:20480\n\t"
        "ds_read_b64_tr_b16 a[198:199], v205 offset:20992\n\t"
        "ds_read_b64_tr_b16 a[200:201], v206 offset:20480\n\t"
        "ds_read_b64_tr_b16 a[202:203], v206 offset:20992\n\t"
        "ds_read_b64_tr_b16 a[204:205], v207 offset:20480\n\t"
        "ds_read_b64_tr_b16 a[206:207], v207 offset:20992\n\t"
        "ds_read_b128 v[68:71], v204 offset:28672\n\t"
        "ds_read_b64_tr_b16 a[208:209], %[rB0] offset:24576\n\t"
        "ds_read_b64_tr_b16 a[210:211], %[rB0] offset:25088\n\t"
        "ds_read_b64_tr_b16 a[212:213], v205 offset:24576\n\t"
        "ds_read_b64_tr_b16 a[214:215], v205 offset:25088\n\t"
        "ds_read_b64_tr_b16 a[216:217], v206 offset:24576\n\t"
        "ds_read_b64_tr_b16 a[218:219], v206 offset:25088\n\t"
        "ds_read_b64_tr_b16 a[220:221], v207 offset:24576\n\t"
        "ds_read_b64_tr_b16 a[222:223], v207 offset:25088\n\t"
        "s_add_u32 m0, %[ldsw], 0\n\t"
        "s_add_u32 s61, %[rotX], 0\n\t"
        "s_and_b32 s61, s61, 4095\n\t"
        "s_add_u32 s61, s61, 65536\n\t"
        "buffer_load_dwordx4 %[vA0], %[srdX], s61 offen lds\n\t"
        "buffer_load_dwordx4 %[vA1], %[srdX], s61 offen offset:1024 lds\n\t"
        "s_add_u32 m0, %[ldsw], 2048\n\t"
        "s_add_u32 s60, %[rotX], 512\n\t"
        "s_and_b32 s60, s60, 4095\n\t"
        "s_add_u32 s60, s60, 65536\n\t"
        "buffer_load_dwordx4 %[vA0], %[srdX], s60 offen lds\n\t"
        "buffer_load_dwordx4 %[vA1], %[srdX], s60 offen offset:1024 lds\n\t"
        "s_waitcnt lgkmcnt(0)\n\t"
        "s_add_u32 m0, %[ldsw], 4096\n\t"
        "s_add_u32 s61, %[rotX], 1024\n\t"
        "s_and_b32 s61, s61, 4095\n\t"
        "s_add_u32 s61, s61, 65536\n\t"
        "buffer_load_dwordx4 %[vA0], %[srdX], s61 offen lds\n\t"
        "v_mfma_f32_16x16x32_bf16 v[80:83], a[192:195], v[64:67], v[80:83]\n\t"
        "v_mfma_f32_16x16x32_bf16 v[84:87], a[196:199], v[64:67], v[84:87]\n\t"
        "v_mfma_f32_16x16x32_bf16 v[88:91], a[200:203], v[64:67], v[88:91]\n\t"
        "buffer_load_dwordx4 %[vA1], %[srdX], s61 offen offset:1024 lds\n\t"
        "v_mfma_f32_16x16x32_bf16 v[92:95], a[204:207], v[64:67], v[92:95]\n\t"
        "v_mfma_f32_16x16x32_bf16 v[80:83], a[208:211], v[68:71], v[80:83]\n\t"
        "v_mfma_f32_16x16x32_bf16 v[84:87], a[212:215], v[68:71], v[84:87]\n\t"
        "s_add_u32 m0, %[ldsw], 6144\n\t"
        "s_add_u32 s60, %[rotX], 1536\n\t"
        "s_and_b32 s60, s60, 4095\n\t"
        "s_add_u32 s60, s60, 65536\n\t"
        "buffer_load_dwordx4 %[vA0], %[srdX], s60 offen lds\n\t"
        "v_mfma_f32_16x16x32_bf16 v[88:91], a[216:219], v[68:71], v[88:91]\n\t"
        "v_mfma_f32_16x16x32_bf16 v[92:95], a[220:223], v[68:71], v[92:95]\n\t"
        "buffer_load_dwordx4 %[vA1], %[srdX], s60 offen offset:1024 lds\n\t"
        "s_waitcnt vmcnt(8)\n\t"
        "ds_read_b128 v[72:75], %[rA0] offset:38912\n\t"
        "ds_read_b64_tr_b16 a[224:225], %[rB0] offset:30720\n\t"
        "ds_read_b64_tr_b16 a[226:227], %[rB0] offset:31232\n\t"
        "ds_read_b64_tr_b16 a[228:229], v205 offset:30720\n\t"
        "ds_read_b64_tr_b16 a[230:231], v205 offset:31232\n\t"
        "ds_read_b64_tr_b16 a[232:233], v206 offset:30720\n\t"
        "ds_read_b64_tr_b16 a[234:235], v206 offset:31232\n\t"
        "ds_read_b64_tr_b16 a[236:237], v207 offset:30720\n\t"
        "ds_read_b64_tr_b16 a[238:239], v207 offset:31232\n\t"
        "ds_read_b128 v[76:79], v204 offset:38912\n\t"
        "ds_read_b64_tr_b16 a[240:241], %[rB0] offset:34816\n\t"
        "ds_read_b64_tr_b16 a[242:243], %[rB0] offset:35328\n\t"
        "ds_read_b64_tr_b16 a[244:245], v205 offset:34816\n\t"
        "ds_read_b64_tr_b16 a[246:247], v205 offset:35328\n\t"
        "ds_read_b64_tr_b16 a[248:249], v206 offset:34816\n\t"
        "ds_read_b64_tr_b16 a[250:251], v206 offset:35328\n\t"
        "ds_read_b64_tr_b16 a[252:253], v207 offset:34816\n\t"
        "ds_read_b64_tr_b16 a[254:255], v207 offset:35328\n\t"
        "s_waitcnt lgkmcnt(9)\n\t"
        "v_mfma_f32_16x16x32_bf16 v[80:83], a[224:227], v[72:75], v[80:83]\n\t"
        "v_mfma_f32_16x16x32_bf16 v[84:87], a[228:231], v[72:75], v[84:87]\n\t"
        "v_mfma_f32_16x16x32_bf16 v[88:91], a[232:235], v[72:75], v[88:91]\n\t"
        "v_mfma_f32_16x16x32_bf16 v[92:95], a[236:239], v[72:75], v[92:95]\n\t"
        "s_waitcnt lgkmcnt(0)\n\t"
        "v_mfma_f32_16x16x32_bf16 v[80:83], a[240:243], v[76:79], v[80:83]\n\t"
        "v_mfma_f32_16x16x32_bf16 v[84:87], a[244:247], v[76:79], v[84:87]\n\t"
        "v_mfma_f32_16x16x32_bf16 v[88:91], a[248:251], v[76:79], v[88:91]\n\t"
        "v_mfma_f32_16x16x32_bf16 v[92:95], a[252:255], v[76:79], v[92:95]\n\t"
        "s_nop 7\n\t"
        "s_nop 1\n\t"
        "ds_write_b128 %[vW], v[80:83] offset:20480\n\t"
        "ds_write_b128 %[vW], v[84:87] offset:21504\n\t"
        "ds_write_b128 v208, v[88:91] offset:20480\n\t"
        "ds_write_b128 v208, v[92:95] offset:21504\n\t"
        "s_waitcnt lgkmcnt(0)\n\t"
        "s_barrier\n\t"
        "s_add_u32 m0, %[ldsw], 10240\n\t"
        "s_add_u32 s61, %[rotX], 2048\n\t"
        "s_and_b32 s61, s61, 4095\n\t"
        "s_add_u32 s61, s61, 65536\n\t"
        "buffer_load_dwordx4 %[vA0], %[srdX], s61 offen lds\n\t"
        "buffer_load_dwordx4 %[vA1], %[srdX], s61 offen offset:1024 lds\n\t"
        "s_add_u32 m0, %[ldsw], 12288\n\t"
        "s_add_u32 s60, %[rotX], 2560\n\t"
        "s_and_b32 s60, s60, 4095\n\t"
        "s_add_u32 s60, s60, 65536\n\t"
        "buffer_load_dwordx4 %[vA0], %[srdX], s60 offen lds\n\t"
        "buffer_load_dwordx4 %[vA1], %[srdX], s60 offen offset:1024 lds\n\t"
        "s_add_u32 m0, %[ldsw], 14336\n\t"
        "s_add_u32 s61, %[rotX], 3072\n\t"
        "s_and_b32 s61, s61, 4095\n\t"
        "s_add_u32 s61, s61, 65536\n\t"
        "buffer_load_dwordx4 %[vA0], %[srdX], s61 offen lds\n\t"
        "buffer_load_dwordx4 %[vA1], %[srdX], s61 offen offset:1024 lds\n\t"
        "s_add_u32 m0, %[ldsw], 16384\n\t"
        "s_add_u32 s60, %[rotX], 3584\n\t"
        "s_and_b32 s60, s60, 4095\n\t"
        "s_add_u32 s60, s60, 65536\n\t"
        "buffer_load_dwordx4 %[vA0], %[srdX], s60 offen lds\n\t"
        "buffer_load_dwordx4 %[vA1], %[srdX], s60 offen offset:1024 lds\n\t"
        "s_waitcnt vmcnt(8)\n\t"
        "ds_read_b128 v[112:115], %[rA0] offset:0\n\t"
        "ds_read_b128 v[116:119], v204 offset:0\n\t"
        "ds_read_b128 v[120:123], %[rA0] offset:2048\n\t"
        "ds_read_b128 v[124:127], v204 offset:2048\n\t"
        "ds_read_b128 v[128:131], %[rA0] offset:4096\n\t"
        "ds_read_b128 v[132:135], v204 offset:4096\n\t"
        "ds_read_b128 v[136:139], %[rA0] offset:6144\n\t"
        "ds_read_b128 v[140:143], v204 offset:6144\n\t"
        "s_waitcnt lgkmcnt(6)\n\t"
        "v_mfma_f32_16x16x32_bf16 v[96:99], a[0:3], v[112:115], 0\n\t"
        "v_mfma_f32_16x16x32_bf16 v[100:103], a[4:7], v[112:115], 0\n\t"
        "v_mfma_f32_16x16x32_bf16 v[104:107], a[8:11], v[112:115], 0\n\t"
        "v_mfma_f32_16x16x32_bf16 v[108:111], a[12:15], v[112:115], 0\n\t"
        "v_mfma_f32_16x16x32_bf16 v[96:99], a[16:19], v[116:119], v[96:99]\n\t"
        "v_mfma_f32_16x16x32_bf16 v[100:103], a[20:23], v[116:119], v[100:103]\n\t"
        "v_mfma_f32_16x16x32_bf16 v[104:107], a[24:27], v[116:119], v[104:107]\n\t"
        "v_mfma_f32_16x16x32_bf16 v[108:111], a[28:31], v[116:119], v[108:111]\n\t"
        "s_waitcnt lgkmcnt(4)\n\t"
        "v_mfma_f32_16x16x32_bf16 v[96:99], a[32:35], v[120:123], v[96:99]\n\t"
        "v_mfma_f32_16x16x32_bf16 v[100:103], a[36:39], v[120:123], v[100:103]\n\t"
        "v_mfma_f32_16x16x32_bf16 v[104:107], a[40:43], v[120:123], v[104:107]\n\t"
        "v_mfma_f32_16x16x32_bf16 v[108:111], a[44:47], v[120:123], v[108:111]\n\t"
        "v_mfma_f32_16x16x32_bf16 v[96:99], a[48:51], v[124:127], v[96:99]\n\t"
        "v_mfma_f32_16x16x32_bf16 v[100:103], a[52:55], v[124:127], v[100:103]\n\t"
        "v_mfma_f32_16x16x32_bf16 v[104:107], a[56:59], v[124:127], v[104:107]\n\t"
        "v_mfma_f32_16x16x32_bf16 v[108:111], a[60:63], v[124:127], v[108:111]\n\t"
        "s_waitcnt lgkmcnt(2)\n\t"
        "v_mfma_f32_16x16x32_bf16 v[96:99], a[64:67], v[128:131], v[96:99]\n\t"
        "v_mfma_f32_16x16x32_bf16 v[100:103], a[68:71], v[128:131], v[100:103]\n\t"
        "v_mfma_f32_16x16x32_bf16 v[104:107], a[72:75], v[128:131], v[104:107]\n\t"
        "v_mfma_f32_16x16x32_bf16 v[108:111], a[76:79], v[128:131], v[108:111]\n\t"
        "v_mfma_f32_16x16x32_bf16 v[96:99], a[80:83], v[132:135], v[96:99]\n\t"
        "v_mfma_f32_16x16x32_bf16 v[100:103], a[84:87], v[132:135], v[100:103]\n\t"
        "v_mfma_f32_16x16x32_bf16 v[104:107], a[88:91], v[132:135], v[104:107]\n\t"
        "v_mfma_f32_16x16x32_bf16 v[108:111], a[92:95], v[132:135], v[108:111]\n\t"
        "s_waitcnt lgkmcnt(0)\n\t"
        "v_mfma_f32_16x16x32_bf16 v[96:99], a[96:99], v[136:139], v[96:99]\n\t"
        "v_mfma_f32_16x16x32_bf16 v[100:103], a[100:103], v[136:139], v[100:103]\n\t"
        "v_mfma_f32_16x16x32_bf16 v[104:107], a[104:107], v[136:139], v[104:107]\n\t"
        "v_mfma_f32_16x16x32_bf16 v[108:111], a[108:111], v[136:139], v[108:111]\n\t"
        "v_mfma_f32_16x16x32_bf16 v[96:99], a[112:115], v[140:143], v[96:99]\n\t"
        "v_mfma_f32_16x16x32_bf16 v[100:103], a[116:119], v[140:143], v[100:103]\n\t"
        "v_mfma_f32_16x16x32_bf16 v[104:107], a[120:123], v[140:143], v[104:107]\n\t"
        "v_mfma_f32_16x16x32_bf16 v[108:111], a[124:127], v[140:143], v[108:111]\n\t"
        "s_waitcnt vmcnt(0)\n\t"
        "ds_read_b128 v[144:147], %[rA0] offset:10240\n\t"
        "ds_read_b128 v[148:151], v204 offset:10240\n\t"
        "ds_read_b128 v[152:155], %[rA0] offset:12288\n\t"
        "ds_read_b128 v[156:159], v204 offset:12288\n\t"
        "ds_read_b128 v[160:163], %[rA0] offset:14336\n\t"
        "ds_read_b128 v[164:167], v204 offset:14336\n\t"
        "ds_read_b128 v[168:171], %[rA0] offset:16384\n\t"
        "ds_read_b128 v[172:175], v204 offset:16384\n\t"
        "s_waitcnt lgkmcnt(6)\n\t"
        "v_mfma_f32_16x16x32_bf16 v[96:99], a[128:131], v[144:147], v[96:99]\n\t"
        "v_mfma_f32_16x16x32_bf16 v[100:103], a[132:135], v[144:147], v[100:103]\n\t"
        "v_mfma_f32_16x16x32_bf16 v[104:107], a[136:139], v[144:147], v[104:107]\n\t"
        "v_mfma_f32_16x16x32_bf16 v[108:111], a[140:143], v[144:147], v[108:111]\n\t"
        "v_mfma_f32_16x16x32_bf16 v[96:99], a[144:147], v[148:151], v[96:99]\n\t"
        "v_mfma_f32_16x16x32_bf16 v[100:103], a[148:151], v[148:151], v[100:103]\n\t"
        "v_mfma_f32_16x16x32_bf16 v[104:107], a[152:155], v[148:151], v[104:107]\n\t"
        "v_mfma_f32_16x16x32_bf16 v[108:111], a[156:159], v[148:151], v[108:111]\n\t"
        "s_waitcnt lgkmcnt(4)\n\t"
        "v_mfma_f32_16x16x32_bf16 v[96:99], a[160:163], v[152:155], v[96:99]\n\t"
        "v_mfma_f32_16x16x32_bf16 v[100:103], a[164:167], v[152:155], v[100:103]\n\t"
        "v_mfma_f32_16x16x32_bf16 v[104:107], a[168:171], v[152:155], v[104:107]\n\t"
        "v_mfma_f32_16x16x32_bf16 v[108:111], a[172:175], v[152:155], v[108:111]\n\t"
        "v_mfma_f32_16x16x32_bf16 v[96:99], a[176:179], v[156:159], v[96:99]\n\t"
        "v_mfma_f32_16x16x32_bf16 v[100:103], a[180:183], v[156:159], v[100:103]\n\t"
        "v_mfma_f32_16x16x32_bf16 v[104:107], a[184:187], v[156:159], v[104:107]\n\t"
        "v_mfma_f32_16x16x32_bf16 v[108:111], a[188:191], v[156:159], v[108:111]\n\t"
        "s_waitcnt lgkmcnt(2)\n\t"
        "v_mfma_f32_16x16x32_bf16 v[96:99], a[192:195], v[160:163], v[96:99]\n\t"
        "v_mfma_f32_16x16x32_bf16 v[100:103], a[196:199], v[160:163], v[100:103]\n\t"
        "v_mfma_f32_16x16x32_bf16 v[104:107], a[200:203], v[160:163], v[104:107]\n\t"
        "v_mfma_f32_16x16x32_bf16 v[108:111], a[204:207], v[160:163], v[108:111]\n\t"
        "v_mfma_f32_16x16x32_bf16 v[96:99], a[208:211], v[164:167], v[96:99]\n\t"
        "v_mfma_f32_16x16x32_bf16 v[100:103], a[212:215], v[164:167], v[100:103]\n\t"
        "v_mfma_f32_16x16x32_bf16 v[104:107], a[216:219], v[164:167], v[104:107]\n\t"
        "v_mfma_f32_16x16x32_bf16 v[108:111], a[220:223], v[164:167], v[108:111]\n\t"
        "s_waitcnt lgkmcnt(0)\n\t"
        "v_mfma_f32_16x16x32_bf16 v[96:99], a[224:227], v[168:171], v[96:99]\n\t"
        "v_mfma_f32_16x16x32_bf16 v[100:103], a[228:231], v[168:171], v[100:103]\n\t"
        "v_mfma_f32_16x16x32_bf16 v[104:107], a[232:235], v[168:171], v[104:107]\n\t"
        "v_mfma_f32_16x16x32_bf16 v[108:111], a[236:239], v[168:171], v[108:111]\n\t"
        "v_mfma_f32_16x16x32_bf16 v[96:99], a[240:243], v[172:175], v[96:99]\n\t"
        "v_mfma_f32_16x16x32_bf16 v[100:103], a[244:247], v[172:175], v[100:103]\n\t"
        "v_mfma_f32_16x16x32_bf16 v[104:107], a[248:251], v[172:175], v[104:107]\n\t"
        "v_mfma_f32_16x16x32_bf16 v[108:111], a[252:255], v[172:175], v[108:111]\n\t"
        "s_nop 7\n\t"
        "s_nop 1\n\t"
        "ds_write_b128 v208, v[104:107] offset:30720\n\t"
        "ds_write_b128 v208, v[108:111] offset:31744\n\t"
        "s_waitcnt lgkmcnt(0)\n\t"
        "s_barrier\n\t"
        "ds_read_b128 v[176:179], %[vR1] offset:30720\n\t"
        "ds_read_b128 v[180:183], %[vR1] offset:31744\n\t"
        "ds_read_b128 v[184:187], %[vR2] offset:30720\n\t"
        "ds_read_b128 v[188:191], %[vR2] offset:31744\n\t"
        "ds_read_b128 v[192:195], %[vR3] offset:30720\n\t"
        "ds_read_b128 v[196:199], %[vR3] offset:31744\n\t"
        "s_waitcnt lgkmcnt(0)\n\t"
        "v_pk_add_f32 v[96:97], v[96:97], v[176:177]\n\t"
        "v_pk_add_f32 v[184:185], v[184:185], v[192:193]\n\t"
        "v_pk_add_f32 v[98:99], v[98:99], v[178:179]\n\t"
        "v_pk_add_f32 v[186:187], v[186:187], v[194:195]\n\t"
        "v_pk_add_f32 v[100:101], v[100:101], v[180:181]\n\t"
        "v_pk_add_f32 v[188:189], v[188:189], v[196:197]\n\t"
        "v_pk_add_f32 v[102:103], v[102:103], v[182:183]\n\t"
        "v_pk_add_f32 v[190:191], v[190:191], v[198:199]\n\t"
        "v_pk_add_f32 v[96:97], v[96:97], v[184:185]\n\t"
        "v_pk_add_f32 v[98:99], v[98:99], v[186:187]\n\t"
        "v_pk_add_f32 v[100:101], v[100:101], v[188:189]\n\t"
        "v_pk_add_f32 v[102:103], v[102:103], v[190:191]\n\t"
        "s_nop 0\n\t"
        "v_cvt_pk_bf16_f32 v200, v96, v97\n\t"
        "v_cvt_pk_bf16_f32 v201, v98, v99\n\t"
        "v_cvt_pk_bf16_f32 v202, v100, v101\n\t"
        "v_cvt_pk_bf16_f32 v203, v102, v103\n\t"
        "s_nop 1\n\t"
        "v_permlane32_swap_b32 v200, v202\n\t"
        "v_permlane32_swap_b32 v201, v203\n\t"
        "s_nop 0\n\t"
        "v_permlane16_swap_b32 v200, v202\n\t"
        "v_permlane16_swap_b32 v201, v203\n\t"
        "global_store_dwordx4 v209, v[200:203], %[outb] sc0 sc1\n\t"
        "s_nop 1\n\t"
        "s_waitcnt vmcnt(0)\n\t"
        "21:\n\t"
        "s_barrier\n\t"
        "s_mov_b32 s62, 0\n\t"
        "s_cmp_eq_u32 %[rem], 0\n\t"
        "s_cbranch_scc1 9f\n\t"
        "s_cmp_eq_u32 %[wave], 3\n\t"
        "s_cbranch_scc0 2f\n\t"
        "s_mov_b64 exec, 1\n\t"
        "v_mov_b32 v211, 0\n\t"
        "v_add_u32 v212, 1, %[call]\n\t"
        "global_store_dword v211, v212, %[flagp] sc0 sc1\n\t"
        "s_mov_b64 exec, -1\n\t"
        "s_branch 9f\n\t"
        "2:\n\t"
        "s_cmp_eq_u32 %[wave], 0\n\t"
        "s_cbranch_scc0 9f\n\t"
        "s_mov_b64 exec, 1\n\t"
        "v_mov_b32 v211, 0\n\t"
        "v_add_u32 v212, 1, %[call]\n\t"
        "s_memrealtime s[92:93]\n\t"
        "s_mov_b32 s63, 0\n\t"
        "4:\n\t"
        "global_load_dword v176, v211, %[inp] sc0 sc1\n\t"
        "s_sleep 4\n\t"
        "global_load_dword v177, v211, %[inp] sc0 sc1\n\t"
        "s_sleep 4\n\t"
        "global_load_dword v178, v211, %[inp] sc0 sc1\n\t"
        "s_sleep 4\n\t"
        "global_load_dword v179, v211, %[inp] sc0 sc1\n\t"
        "5:\n\t"
        "s_waitcnt vmcnt(3)\n\t"
        "v_cmp_ge_u32 vcc, v176, v212\n\t"
        "s_cbranch_vccnz 8f\n\t"
        "global_load_dword v176, v211, %[inp] sc0 sc1\n\t"
        "s_waitcnt vmcnt(3)\n\t"
        "v_cmp_ge_u32 vcc, v177, v212\n\t"
        "s_cbranch_vccnz 8f\n\t"
        "global_load_dword v177, v211, %[inp] sc0 sc1\n\t"
        "s_waitcnt vmcnt(3)\n\t"
        "v_cmp_ge_u32 vcc, v178, v212\n\t"
        "s_cbranch_vccnz 8f\n\t"
        "global_load_dword v178, v211, %[inp] sc0 sc1\n\t"
        "s_waitcnt vmcnt(3)\n\t"
        "v_cmp_ge_u32 vcc, v179, v212\n\t"
        "s_cbranch_vccnz 8f\n\t"
        "global_load_dword v179, v211, %[inp] sc0 sc1\n\t"
        "s_add_u32 s63, s63, 1\n\t"
        "s_and_b32 s60, s63, 15\n\t"
        "s_cbranch_scc1 5b\n\t"
        "s_memrealtime s[94:95]\n\t"
        "global_load_dword v213, v211, %[tmo] sc1\n\t"
        "s_waitcnt vmcnt(0) lgkmcnt(0)\n\t"
        "s_sub_u32 s94, s94, s92\n\t"
        "s_subb_u32 s95, s95, s93\n\t"
        "v_cmp_eq_u32 vcc, v213, v212\n\t"
        "s_cbranch_vccnz 6f\n\t"
        "s_cmp_lg_u32 s95, 0\n\t"
        "s_cbranch_scc1 6f\n\t"
        "s_cmp_gt_u32 s94, 40000000\n\t"
        "s_cbranch_scc0 4b\n\t"
        "6:\n\t"
        "global_store_dword v211, v212, %[tmo] sc1\n\t"
        "s_mov_b32 s62, 1\n\t"
        "8:\n\t"
        "s_waitcnt lgkmcnt(0)\n\t"
        "s_mov_b64 exec, -1\n\t"
        "9:\n\t"
        "s_mov_b32 %[bad], s62\n\t"
        : [call] "=&v"(ticket), [bad] "=s"(bad)
        : [vA0] "v"(vA0), [vA1] "v"(vA1), [vB0] "v"(vB0), [vB1] "v"(vB1), [vB2] "v"(vB2), [vB3] "v"(vB3),
          [rA0] "v"(rA0), [rB0] "v"(rB0), [vW] "v"(vW), [vR1] "v"(vR1), [vR2] "v"(vR2), [vR3] "v"(vR3), [vO] "v"(vO),
          [srdX] "s"(srdX), [srdW] "s"(srdW), [ldsw] "s"(ldsw), [rotW] "s"(rotW), [rotX] "s"(rotX), [wave] "s"(wave), [outb] "s"(outb),
          [flagp] "s"(flagp), [rem] "s"(rem), [inp] "s"(inp), [tmo] "s"(tmo), [cnt] "s"(cnt), [tick] "s"(tick)
        :
          "v64", "v65", "v66", "v67", "v68", "v69", "v70", "v71", "v72", "v73", "v74", "v75", "v76", "v77", "v78", "v79",
          "v80", "v81", "v82", "v83", "v84", "v85", "v86", "v87", "v88", "v89", "v90", "v91", "v92", "v93", "v94", "v95",
          "v96", "v97", "v98", "v99", "v100", "v101", "v102", "v103", "v104", "v105", "v106", "v107", "v108", "v109", "v110", "v111",
          "v112", "v113", "v114", "v115", "v116", "v117", "v118", "v119", "v120", "v121", "v122", "v123", "v124", "v125", "v126", "v127",
          "v128", "v129", "v130", "v131", "v132", "v133", "v134", "v135", "v136", "v137", "v138", "v139", "v140", "v141", "v142", "v143",
          "v144", "v145", "v146", "v147", "v148", "v149", "v150", "v151", "v152", "v153", "v154", "v155", "v156", "v157", "v158", "v159",
          "v160", "v161", "v162", "v163", "v164", "v165", "v166", "v167", "v168", "v169", "v170", "v171", "v172", "v173", "v174", "v175",
          "v176", "v177", "v178", "v179", "v180", "v181", "v182", "v183", "v184", "v185", "v186", "v187", "v188", "v189", "v190", "v191",
          "v192", "v193", "v194", "v195", "v196", "v197", "v198", "v199", "v200", "v201", "v202", "v203", "v204", "v205", "v206", "v207",
          "v208", "v209", "v210", "v211", "v212", "v213", "v214", "v215", "a0", "a1", "a2", "a3", "a4", "a5", "a6", "a7",
          "a8", "a9", "a10", "a11", "a12", "a13", "a14", "a15", "a16", "a17", "a18", "a19", "a20", "a21", "a22", "a23",
          "a24", "a25", "a26", "a27", "a28", "a29", "a30", "a31", "a32", "a33", "a34", "a35", "a36", "a37", "a38", "a39",
          "a40", "a41", "a42", "a43", "a44", "a45", "a46", "a47", "a48", "a49", "a50", "a51", "a52", "a53", "a54", "a55",
          "a56", "a57", "a58", "a59", "a60", "a61", "a62", "a63", "a64", "a65", "a66", "a67", "a68", "a69", "a70", "a71",
          "a72", "a73", "a74", "a75", "a76", "a77", "a78", "a79", "a80", "a81", "a82", "a83", "a84", "a85", "a86", "a87",
          "a88", "a89", "a90", "a91", "a92", "a93", "a94", "a95", "a96", "a97", "a98", "a99", "a100", "a101", "a102", "a103",
          "a104", "a105", "a106", "a107", "a108", "a109", "a110", "a111", "a112", "a113", "a114", "a115", "a116", "a117", "a118", "a119",
          "a120", "a121", "a122", "a123", "a124", "a125", "a126", "a127", "a128", "a129", "a130", "a131", "a132", "a133", "a134", "a135",
          "a136", "a137", "a138", "a139", "a140", "a141", "a142", "a143", "a144", "a145", "a146", "a147", "a148", "a149", "a150", "a151",
          "a152", "a153", "a154", "a155", "a156", "a157", "a158", "a159", "a160", "a161", "a162", "a163", "a164", "a165", "a166", "a167",
          "a168", "a169", "a170", "a171", "a172", "a173", "a174", "a175", "a176", "a177", "a178", "a179", "a180", "a181", "a182", "a183",
          "a184", "a185", "a186", "a187", "a188", "a189", "a190", "a191", "a192", "a193", "a194", "a195", "a196", "a197", "a198", "a199",
          "a200", "a201", "a202", "a203", "a204", "a205", "a206", "a207", "a208", "a209", "a210", "a211", "a212", "a213", "a214", "a215",
          "a216", "a217", "a218", "a219", "a220", "a221", "a222", "a223", "a224", "a225", "a226", "a227", "a228", "a229", "a230", "a231",
          "a232", "a233", "a234", "a235", "a236", "a237", "a238", "a239", "a240", "a241", "a242", "a243", "a244", "a245", "a246", "a247",
          "a248", "a249", "a250", "a251", "a252", "a253", "a254", "a255", "s60", "s61", "s62", "s63", "s92", "s93", "s94", "s95",
          "vcc", "scc", "memory");

    if (bad != 0 && lane == 0) {
        GAS unsigned short* e = (GAS unsigned short*)p.myout + (size_t)(p.rank * MLOC + m0) * NLOC + nl0;
        __hip_atomic_store(e, (unsigned short)0x7FC0, __ATOMIC_RELAXED, __HIP_MEMORY_SCOPE_SYSTEM);
    }
}

}

extern "C" void kernel_launch(int rank, int nranks, ncclComm_t comm, hipStream_t stream,
                              void* const* d_in, const int* in_sizes, int n_in,
                              void* d_out, int out_size,
                              const int* in_dtypes, int out_dtype,
                              void* const* d_out_all, void* const* d_ws_all, size_t ws_bytes) {
    (void)comm; (void)in_dtypes; (void)out_dtype;
    if (nranks != NRANK || (unsigned)rank >= (unsigned)NRANK || n_in != 2 || in_sizes[0] != MLOC * KDIM ||
        in_sizes[1] != KDIM * NDIM || out_size != NDIM * NLOC || ws_bytes < WS_END) {
        fprintf(stderr, "kernel_launch: unexpected shapes\n");
        return;
    }
    Params p;
    p.x = d_in[0];
    p.w = d_in[1];
    p.myws = (char*)d_ws_all[rank];
    p.myout = (char*)d_out;
    for (int i = 0; i < NRANK; ++i) { p.out[i] = d_out_all[i]; p.ws[i] = d_ws_all[i]; }
    p.rank = rank;
    p.pad = 0;
    hipLaunchKernelGGL(gemm_a2a, dim3(NWG), dim3(256), 0, stream, p);
    HIP_CHECK(hipGetLastError());
}
```
